```python
import numpy as np

kernel_name = "gpu_a2a_gemm_m4096_k4096_n16384_e5m2e5m2_dqsilu"


ABSMAX_RTOL = 0.05

COMPUTE_DTYPE = "float32"
EPILOGUE = "dequant_silu"
IN_QUANT = None
OUT_QUANT = None
IN_FP8 = "e5m2e5m2"
IN_DTYPES = {"x": "e5m2", "w_mat": "e5m2",
             "scale_x": "float32", "scale_w": "float32"}
WORLD_SIZE = 8

def _snap_fp8(x, mant_bits, max_abs, min_normal):
    a = np.abs(x.astype(np.float64))
    _m, _e = np.frexp(a)
    q = np.where(a >= min_normal, np.ldexp(1.0, _e - 1 - mant_bits),
                 min_normal * 2.0 ** -mant_bits)
    snapped = np.minimum(np.round(a / q) * q, max_abs)
    return (np.sign(x) * snapped).astype(np.float32)

def setup_inputs():

    x = _snap_fp8(np.random.randn(4096, 4096).astype(np.float32), 2, 57344.0, 2.0 ** -14)
    w_mat = _snap_fp8(np.random.randn(4096, 16384).astype(np.float32), 2, 57344.0, 2.0 ** -14)
    scale_x = (0.02 * (1.0 + np.random.rand(1))).astype(np.float32)
    scale_w = (0.02 * (1.0 + np.random.rand(1))).astype(np.float32)
    return {"x": x, "w_mat": w_mat, "scale_x": scale_x, "scale_w": scale_w}

def reference(x, w_mat, scale_x, scale_w):
    acc = (x @ w_mat).astype(np.float32)
    y = (acc * (scale_x * scale_w)).astype(np.float32)
    return (y / (1.0 + np.exp(-np.clip(y, -60.0, 60.0)))).astype(np.float32)
```

```cpp
#include <hip/hip_runtime.h>
#include <rccl/rccl.h>
#include <cstdio>

#define HIP_CHECK(expr) do { hipError_t _e = (expr); if (_e != hipSuccess) { \
    fprintf(stderr, "HIP error %s at %s:%d\n", hipGetErrorString(_e), __FILE__, __LINE__); \
    return; } } while (0)

#define LAS __attribute__((address_space(3)))
typedef int v2i __attribute__((ext_vector_type(2)));
typedef int v4i __attribute__((ext_vector_type(4)));
typedef int v8i __attribute__((ext_vector_type(8)));
typedef unsigned v2u __attribute__((ext_vector_type(2)));
typedef unsigned v4u __attribute__((ext_vector_type(4)));
typedef float v16f __attribute__((ext_vector_type(16)));

constexpr int NR = 8;
constexpr int MS = 512, N = 16384, K = 4096;
constexpr int KS = K / NR;
constexpr int BLK = MS * KS;
constexpr int SLOT = 49152, XB = 32768;
constexpr int NSLOT = 3, NSTEP = 32;
constexpr int LDS_CTRL = NSLOT * SLOT;
constexpr int LDS_BYTES = LDS_CTRL + 64;
constexpr size_t WS_CTR = 0;
constexpr size_t WS_TMO = 16384;
constexpr size_t WS_FLAG = 32768;
constexpr size_t WS_X = 1 << 20;
constexpr size_t WS_NEED = WS_X + 2 * (size_t)NR * BLK;
constexpr unsigned long long DEADLINE_TICKS = 100000000ull;

struct Params {
  const char* x; const char* w; const float* sx; const float* sw; float* out;
  char* ws[NR];
  int rank; int pad;
};

__device__ __forceinline__ v8i cat4(v4i lo, v4i hi) { return __builtin_shufflevector(lo, hi, 0, 1, 2, 3, 4, 5, 6, 7); }
__device__ __forceinline__ v8i cat2(v2i a, v2i b, v2i c, v2i d) {
  v4i lo = __builtin_shufflevector(a, b, 0, 1, 2, 3), hi = __builtin_shufflevector(c, d, 0, 1, 2, 3);
  return cat4(lo, hi);
}
#define WAIT_LGKM0() __builtin_amdgcn_s_waitcnt(0xC07F)
#define BAR() __builtin_amdgcn_s_barrier()
#define SCHED() __builtin_amdgcn_sched_barrier(0)

__device__ __forceinline__ void step_pq(int s, int rank, int& p, int& q) {
  if (s < 4) { p = rank; q = s; }
  else { const int u = s - 4; q = u / 7; p = (rank + 1 + (u - q * 7)) & 7; }
}

__global__ __launch_bounds__(512, 2) void a2a_gemm_k(const Params P) {
  extern __shared__ __attribute__((aligned(16))) char smem[];
  const int tid = threadIdx.x, wid = __builtin_amdgcn_readfirstlane(tid >> 6), lane = tid & 63;
  const int b = blockIdx.x, grp = b & 7, idx = b >> 3;
  const int mh = idx & 1, nt = grp * 16 + (idx >> 1);
  const int m0 = mh * 256, n0 = nt * 128;
  const int rank = P.rank;
  char* const myws = P.ws[rank];
  unsigned call;
  {
    unsigned t = 0;
    if (lane == 0) t = __hip_atomic_fetch_add((unsigned*)(myws + WS_CTR) + b * 8 + wid, 1u, __ATOMIC_RELAXED, __HIP_MEMORY_SCOPE_AGENT);
    call = __builtin_amdgcn_readfirstlane(t) + 1u;
  }
  unsigned* const tmo = (unsigned*)(myws + WS_TMO);
  LAS unsigned* const lbad = (LAS unsigned*)((LAS char*)smem + LDS_CTRL);

  if (wid >= 4) {
    const int lw = wid - 4;
    __amdgpu_buffer_rsrc_t rxl = __builtin_amdgcn_make_buffer_rsrc((void*)(P.x + (size_t)rank * BLK), 0, BLK, 0x00020000);
    __amdgpu_buffer_rsrc_t rw = __builtin_amdgcn_make_buffer_rsrc((void*)P.w, 0, K * N, 0x00020000);
    const int r8 = lane >> 3, sl = lane & 7;
    const int vx0 = (m0 + lw * 64 + r8) * KS + ((sl ^ (r8 >> 1)) << 4);
    const int vx1 = (m0 + lw * 64 + r8) * KS + ((sl ^ (4 + (r8 >> 1))) << 4);
    const int vw = (lw * 32 + r8) * N + n0 + ((sl ^ (((r8 >> 1) & 3) << 1)) << 4);
    auto issue_w = [&](int p, int q, int slot) {
      const int sow = (p * KS + q * 128) * N;
      LAS char* lwp = (LAS char*)smem + slot * SLOT + XB + lw * 32 * 128;
#pragma unroll
      for (int i = 0; i < 4; ++i)
        __builtin_amdgcn_raw_ptr_buffer_load_lds(rw, lwp + i * 1024, 16, vw, sow + i * 8 * N, 0, 0);
    };
    auto issue_local = [&](int s, int slot) {
      issue_w(rank, s, slot);
      LAS char* lx = (LAS char*)smem + slot * SLOT + lw * 64 * 128;
#pragma unroll
      for (int i = 0; i < 8; ++i)
        __builtin_amdgcn_raw_ptr_buffer_load_lds(rxl, lx + i * 1024, 16, (i & 1) ? vx1 : vx0, s * 128 + i * 8 * KS, 0, 0);
    };
    issue_local(0, 0); issue_local(1, 1); issue_local(2, 2);
    if (lw == 0 && lane == 0) *lbad = 0u;
    __amdgpu_buffer_rsrc_t rxs = __builtin_amdgcn_make_buffer_rsrc((void*)(myws + WS_X + (size_t)(call & 1u) * NR * BLK), 0, NR * BLK, 0x00020000);
    const unsigned* const fl = (const unsigned*)(myws + WS_FLAG) + (mh * 4 + lw) * 128 + lane * 2;
    unsigned long long ready = 0;
    asm volatile("s_waitcnt vmcnt(24) lgkmcnt(0)" ::: "memory");
    BAR();
    int slot = 0;
    for (int t = 0; t < NSTEP; ++t) {
      if (t + 2 < NSTEP) asm volatile("s_waitcnt vmcnt(12)" ::: "memory");
      else asm volatile("s_waitcnt vmcnt(0)" ::: "memory");
      BAR();
      const int s = t + 3;
      if (s < 4) issue_local(s, slot);
      else if (s < NSTEP) {
        int p, q; step_pq(s, rank, p, q);
        const int bit = 2 * (p * 4 + q);
        if (!((ready >> bit) & 1ull)) {
          const unsigned long long t0 = __builtin_amdgcn_s_memrealtime();
          for (;;) {
            v2u f;
            asm volatile("global_load_dwordx2 %0, %1, off sc0 sc1\n\ts_waitcnt vmcnt(0)" : "=&v"(f) : "v"(fl) : "memory");
            const unsigned long long ok = __ballot(f.x >= call && f.y >= call);
            ready = ok & (ok >> 1);
            if ((ready >> bit) & 1ull) break;
            const unsigned tv = __builtin_amdgcn_readfirstlane(__hip_atomic_load(tmo, __ATOMIC_RELAXED, __HIP_MEMORY_SCOPE_AGENT));
            if (tv == call || __builtin_amdgcn_s_memrealtime() - t0 > DEADLINE_TICKS) {
              if (lane == 0) { __hip_atomic_store(tmo, call, __ATOMIC_RELAXED, __HIP_MEMORY_SCOPE_AGENT); *lbad = 1u; }
              ready = ~0ull;
              break;
            }
            __builtin_amdgcn_s_sleep(2);
          }
        }
        issue_w(p, q, slot);
        LAS char* lx = (LAS char*)smem + slot * SLOT + lw * 64 * 128;
#pragma unroll
        for (int i = 0; i < 8; ++i)
          __builtin_amdgcn_raw_ptr_buffer_load_lds(rxs, lx + i * 1024, 16, (i & 1) ? vx1 : vx0, p * BLK + q * 128 + i * 8 * KS, 0, 17);
      }
      slot = (slot == 2) ? 0 : slot + 1;
    }
    return;
  }

  const bool pushes = (grp != rank);
  v4i px0, px1;
  char* pdst; unsigned* pflag;
  {
    const int off = (idx * 16 + (lane >> 3)) * KS + wid * 128 + (lane & 7) * 16;
    const char* src = P.x + (size_t)grp * BLK + off;
    px0 = *(const v4i*)src; px1 = *(const v4i*)(src + 8 * KS);
    char* dws = P.ws[grp];
    pdst = dws + WS_X + (size_t)(call & 1u) * NR * BLK + (size_t)rank * BLK + off;
    pflag = (unsigned*)(dws + WS_FLAG) + (((idx >> 4) * 4 + ((idx >> 2) & 3)) * 32 + rank * 4 + wid) * 4 + (idx & 3);
  }
  auto push_stores = [&]() {
    asm volatile("global_store_dwordx4 %0, %1, off sc0 sc1\n\ts_nop 1" :: "v"(pdst), "v"(px0) : "memory");
    asm volatile("global_store_dwordx4 %0, %1, off sc0 sc1\n\ts_nop 1" :: "v"(pdst + 8 * KS), "v"(px1) : "memory");
  };
  const int t_store = wid == 0 ? -1 : (wid == 1 ? 0 : 2 * wid - 2);
  const int t_flag = wid == 0 ? 1 : 2 * wid + 1;
  if (pushes && wid == 0) push_stores();

  const int wm = wid >> 1, wn = wid & 1;
  const int h = lane >> 5, l31 = lane & 31;
  const int fx = (l31 >> 1) & 7;
  unsigned ax[2][2];
#pragma unroll
  for (int kk = 0; kk < 2; ++kk)
#pragma unroll
    for (int j = 0; j < 2; ++j) ax[kk][j] = (wm * 128 + l31) * 128 + (((4 * kk + 2 * h + j) ^ fx) << 4);
  const int q8 = (lane & 15) >> 1, g1 = (lane >> 4) & 1;
  unsigned aw[2];
#pragma unroll
  for (int j = 0; j < 2; ++j) aw[j] = XB + (32 * h + q8) * 128 + (((4 * wn + 2 * j + g1) ^ (((q8 >> 1) & 3) << 1)) << 4) + 8 * (lane & 1);
  v16f acc[4][2];
#pragma unroll
  for (int i = 0; i < 4; ++i)
#pragma unroll
    for (int j = 0; j < 2; ++j) acc[i][j] = (v16f){};
  v8i a0[4], b0[2], a1[4], b1[2];
  const unsigned sb0 = (unsigned)(size_t)(LAS char*)smem;
#define RDA(dst, i, kk, sb) dst[i] = cat4(*(LAS v4i*)(size_t)(ax[kk][0] + (sb) + (i) * 4096), *(LAS v4i*)(size_t)(ax[kk][1] + (sb) + (i) * 4096))
#define TR(j, kk, t, sb) __builtin_amdgcn_ds_read_tr8_b64_v2i32((LAS v2i*)(size_t)(aw[j] + (sb) + (kk) * 8192 + (t) * 1024))
#define RDB(dst, j, kk, sb) dst[j] = cat2(TR(j, kk, 0, sb), TR(j, kk, 1, sb), TR(j, kk, 2, sb), TR(j, kk, 3, sb))
#define MM(i, j, A, B) acc[i][j] = __builtin_amdgcn_mfma_scale_f32_32x32x64_f8f6f4(A[i], B[j], acc[i][j], 1, 1, 0, 0, 0, 0)
#define SGB(m, n) __builtin_amdgcn_sched_group_barrier(m, n, 0)
#define PATTERN() do { SGB(0x8,1); SGB(0x100,2); SGB(0x8,1); SGB(0x100,2); SGB(0x8,1); SGB(0x100,3); SGB(0x8,1); SGB(0x100,3); \
                       SGB(0x8,1); SGB(0x100,3); SGB(0x8,1); SGB(0x100,3); SGB(0x8,1); SGB(0x8,1); } while (0)
  BAR();
  asm volatile("" : "+v"(px0), "+v"(px1));
  {
    const unsigned sb = sb0;
    RDB(b0, 0, 0, sb); RDA(a0, 0, 0, sb); RDB(b0, 1, 0, sb); RDA(a0, 1, 0, sb); RDA(a0, 2, 0, sb); RDA(a0, 3, 0, sb);
  }
  unsigned sb = sb0;
  int slot = 0;
  unsigned tmo_seen = 0;
  for (int t = 0; t < NSTEP; ++t) {
    const int nslot = (slot == 2) ? 0 : slot + 1;
    const unsigned sbn = sb0 + nslot * SLOT;
    if (pushes) {
      if (t == t_store) push_stores();
      if (t == t_flag) {
        asm volatile("s_waitcnt vmcnt(0)" ::: "memory");
        if (lane == 0) __hip_atomic_store(pflag, call, __ATOMIC_RELAXED, __HIP_MEMORY_SCOPE_SYSTEM);
      }
    }
    SCHED();
    RDB(b1, 0, 1, sb); RDA(a1, 0, 1, sb); RDB(b1, 1, 1, sb); RDA(a1, 1, 1, sb); RDA(a1, 2, 1, sb); RDA(a1, 3, 1, sb);
    MM(0, 0, a0, b0); MM(0, 1, a0, b0); MM(1, 0, a0, b0); MM(1, 1, a0, b0); MM(2, 0, a0, b0); MM(2, 1, a0, b0); MM(3, 0, a0, b0); MM(3, 1, a0, b0);
    PATTERN();
    SCHED();
    WAIT_LGKM0();
    BAR();
    SCHED();
    RDB(b0, 0, 0, sbn); RDA(a0, 0, 0, sbn); RDB(b0, 1, 0, sbn); RDA(a0, 1, 0, sbn); RDA(a0, 2, 0, sbn); RDA(a0, 3, 0, sbn);
    MM(0, 0, a1, b1); MM(0, 1, a1, b1); MM(1, 0, a1, b1); MM(1, 1, a1, b1); MM(2, 0, a1, b1); MM(2, 1, a1, b1); MM(3, 0, a1, b1); MM(3, 1, a1, b1);
    PATTERN();
    SCHED();
    sb = sbn; slot = nslot;
  }
  tmo_seen = __hip_atomic_load(tmo, __ATOMIC_RELAXED, __HIP_MEMORY_SCOPE_AGENT);
  const bool bad = (tmo_seen == call) || (*(volatile LAS unsigned*)lbad != 0u);
  const float s = bad ? __builtin_nanf("") : P.sx[0] * P.sw[0];
  __amdgpu_buffer_rsrc_t ro = __builtin_amdgcn_make_buffer_rsrc((void*)P.out, 0, MS * N * 4, 0x00020000);
  const int vo = ((m0 + wm * 128 + 4 * h) * N + n0 + wn * 64 + l31) * 4;
#pragma unroll
  for (int i = 0; i < 4; ++i)
#pragma unroll
    for (int r = 0; r < 16; ++r)
#pragma unroll
      for (int j = 0; j < 2; ++j) {
        const float y = acc[i][j][r] * s;
        const float c = __builtin_amdgcn_fmed3f(y, -60.f, 60.f);
        const float e = __builtin_amdgcn_exp2f(c * -1.44269504088896f);
        const float o = y * __builtin_amdgcn_rcpf(1.0f + e);
        __builtin_amdgcn_raw_buffer_store_b32(__builtin_bit_cast(unsigned, o), ro, vo + j * 128, (32 * i + (r & 3) + 8 * (r >> 2)) * N * 4, 16);
      }
}

extern "C" void kernel_launch(int rank, int nranks, ncclComm_t comm,
                              hipStream_t stream,
                              void* const* d_in, const int* in_sizes, int n_in,
                              void* d_out, int out_size,
                              const int* in_dtypes, int out_dtype,
                              void* const* d_out_all,
                              void* const* d_ws_all, size_t ws_bytes) {
  (void)comm; (void)in_dtypes; (void)out_dtype; (void)d_out_all;
  if (nranks != NR || rank < 0 || rank >= NR || n_in != 4 || in_sizes[0] != NR * BLK || in_sizes[1] != K * N ||
      out_size != MS * N || ws_bytes < WS_NEED) {
    fprintf(stderr, "kernel_launch: unexpected problem shape\n");
    return;
  }
  static bool attr_set = false;
  if (!attr_set) {
    HIP_CHECK(hipFuncSetAttribute((const void*)a2a_gemm_k, hipFuncAttributeMaxDynamicSharedMemorySize, LDS_BYTES));
    attr_set = true;
  }
  Params P;
  P.x = (const char*)d_in[0]; P.w = (const char*)d_in[1]; P.sx = (const float*)d_in[2]; P.sw = (const float*)d_in[3];
  P.out = (float*)d_out;
  for (int p = 0; p < NR; ++p) P.ws[p] = (char*)d_ws_all[p];
  P.rank = rank; P.pad = 0;
  a2a_gemm_k<<<256, 512, LDS_BYTES, stream>>>(P);
  HIP_CHECK(hipGetLastError());
}
```

```cpp
#include <hip/hip_runtime.h>
#include <rccl/rccl.h>
#include <cstdio>

#define HIP_CHECK(expr) do { hipError_t _e = (expr); if (_e != hipSuccess) { \
    fprintf(stderr, "HIP error %s at %s:%d\n", hipGetErrorString(_e), __FILE__, __LINE__); \
    return; } } while (0)

#define LAS __attribute__((address_space(3)))
typedef int v2i __attribute__((ext_vector_type(2)));
typedef int v4i __attribute__((ext_vector_type(4)));
typedef int v8i __attribute__((ext_vector_type(8)));
typedef unsigned v4u __attribute__((ext_vector_type(4)));
typedef float v4f __attribute__((ext_vector_type(4)));

constexpr int NR = 8;
constexpr int MS = 512, N = 16384, K = 4096;
constexpr int KS = K / NR;
constexpr int BLK = MS * KS;
constexpr int SLOT = 49152, XB = 32768;
constexpr int NSLOT = 3, NSTEP = 32, NREM = 28;
constexpr int LDS_CTRL = NSLOT * SLOT;
constexpr int LDS_FLG = LDS_CTRL + 256;
constexpr int LDS_BYTES = LDS_FLG + 2048;
constexpr size_t WS_CTR = 0;
constexpr size_t WS_TMO = 32768;
constexpr size_t WS_FLAG = 36864;
constexpr size_t WS_X = 1 << 20;
constexpr size_t WS_NEED = WS_X + 2 * (size_t)NR * BLK;
constexpr unsigned long long DEADLINE_TICKS = 100000000ull;

#define POLL_AHEAD 4
#define T_STORE(w) (2 * ((w) - 1))
#define T_FORCE(w) ((w) == 0 ? 2 : ((w) == 1 ? 8 : 5 + 4 * (w)))
#define PF_DIST 0
#define PF_PRE 0
#define PF_START 0
constexpr int PF_D = PF_DIST;

struct Params {
  const char* x; const char* w; const float* sx; const float* sw; float* out; char* myws;
  char* ws[NR];
  int rank; int pad;
};

#define STAMP(i) do {} while (0)
#define STAMP_FLUSH(n) do {} while (0)

__device__ __forceinline__ v8i cat4(v4i lo, v4i hi) { return __builtin_shufflevector(lo, hi, 0, 1, 2, 3, 4, 5, 6, 7); }
__device__ __forceinline__ v8i cat2(v2i a, v2i b, v2i c, v2i d) {
  v4i lo = __builtin_shufflevector(a, b, 0, 1, 2, 3), hi = __builtin_shufflevector(c, d, 0, 1, 2, 3);
  return cat4(lo, hi);
}
#define WAIT_LGKM0() __builtin_amdgcn_s_waitcnt(0xC07F)
#define BAR() __builtin_amdgcn_s_barrier()
#define SCHED() __builtin_amdgcn_sched_barrier(0)

__device__ __forceinline__ void piece_pq(int u, int rank, int& p, int& q) { q = u / 7; p = (rank + 1 + (u - q * 7)) & 7; }

__global__ __launch_bounds__(512, 2) void a2a_gemm_k(const Params P) {
  extern __shared__ __attribute__((aligned(16))) char smem[];
  const int tid = threadIdx.x, wid = __builtin_amdgcn_readfirstlane(tid >> 6), lane = tid & 63;
  const int b = blockIdx.x, grp = b & 7, idx = b >> 3;
  const int mh = idx & 1, nt = grp * 16 + (idx >> 1);
  const int m0 = mh * 256, n0 = nt * 128;
  const int rank = P.rank;
  char* const myws = P.myws;
  STAMP(0);
  unsigned* const tmo = (unsigned*)(myws + WS_TMO);
  volatile LAS unsigned* const lctl = (volatile LAS unsigned*)((LAS char*)smem + LDS_CTRL);

  if (wid >= 4) {
    const int lw = wid - 4;
    __amdgpu_buffer_rsrc_t rxl = __builtin_amdgcn_make_buffer_rsrc((void*)(P.x + (size_t)rank * BLK), 0, BLK, 0x00020000);
    __amdgpu_buffer_rsrc_t rw = __builtin_amdgcn_make_buffer_rsrc((void*)P.w, 0, K * N, 0x00020000);
    const int r8 = lane >> 3, sl = lane & 7;
    const int vx0 = (m0 + lw * 64 + r8) * KS + ((sl ^ (r8 >> 1)) << 4);
    const int vx1 = (m0 + lw * 64 + r8) * KS + ((sl ^ (4 + (r8 >> 1))) << 4);
    const int vw0 = (lw * 32 + r8) * N + n0 + ((sl ^ ((r8 >> 1) & 3)) << 4);
    const int vw1 = (lw * 32 + r8) * N + n0 + ((sl ^ (((r8 >> 1) & 3) | 4)) << 4);
    auto issue_w = [&](int p, int q, int slot) {
      const int sow = (p * KS + q * 128) * N;
      LAS char* lwp = (LAS char*)smem + slot * SLOT + XB + lw * 32 * 128;
#pragma unroll
      for (int i = 0; i < 4; ++i)
        __builtin_amdgcn_raw_ptr_buffer_load_lds(rw, lwp + i * 1024, 16, (i & 2) ? vw1 : vw0, sow + i * 8 * N, 0, 0);
    };
    auto issue_local = [&](int s, int slot) {
      issue_w(rank, s, slot);
      LAS char* lx = (LAS char*)smem + slot * SLOT + lw * 64 * 128;
#pragma unroll
      for (int i = 0; i < 8; ++i)
        __builtin_amdgcn_raw_ptr_buffer_load_lds(rxl, lx + i * 1024, 16, (i & 1) ? vx1 : vx0, s * 128 + i * 8 * KS, 0, 0);
    };
    issue_local(0, 0);
    BAR();
    issue_local(1, 1); issue_local(2, 2);
    if (lw == 0 && lane < 40 && lane != 1) lctl[lane] = 0u;
    asm volatile("s_waitcnt vmcnt(24) lgkmcnt(0)" ::: "memory");
    BAR();
    STAMP(1);
    const unsigned call = __builtin_amdgcn_readfirstlane(lctl[1]);
    __amdgpu_buffer_rsrc_t rxs = __builtin_amdgcn_make_buffer_rsrc((void*)(myws + WS_X + (size_t)(call & 1u) * NR * BLK), 0, NR * BLK, 0x00020000);
    __amdgpu_buffer_rsrc_t rfl = __builtin_amdgcn_make_buffer_rsrc((void*)(myws + WS_FLAG + mh * 2048), 0, 2048, 0x00020000);
    unsigned flg_addr, my_entry;
    {
      int p, q; piece_pq(min(lane >> 1, NREM - 1), rank, p, q);
      flg_addr = (unsigned)(size_t)((LAS char*)smem + LDS_FLG) + (p * 4 + q) * 64 + (lane & 1) * 32;
      my_entry = 0x100u | (unsigned)(p * 4 + q);
    }
    auto poll_issue = [&]() {
      __builtin_amdgcn_raw_ptr_buffer_load_lds(rfl, (LAS char*)smem + LDS_FLG, 16, lane * 16, 0, 0, 17);
      __builtin_amdgcn_raw_ptr_buffer_load_lds(rfl, (LAS char*)smem + LDS_FLG + 1024, 16, lane * 16, 1024, 0, 17);
    };
    auto poll_eval = [&]() -> unsigned long long {
      const v4u f0 = *(volatile LAS v4u*)(size_t)flg_addr, f1 = *(volatile LAS v4u*)(size_t)(flg_addr + 16);
      const unsigned m = min(min(min(f0.x, f0.y), min(f0.z, f0.w)), min(min(f1.x, f1.y), min(f1.z, f1.w)));
      const unsigned long long ok = __ballot(m >= call && lane < 2 * NREM);
      return ok & (ok >> 1) & 0x5555555555555555ull;
    };
    unsigned long long ready = 0, used = 0;
    constexpr unsigned long long ALL = 0x0055555555555555ull;
    int nsched = 4;
    auto schedule = [&](int upto) {
      unsigned long long avail = ready & ~used;
      if (ready == ALL) upto = NSTEP;
      const int pos = nsched + (int)__builtin_amdgcn_mbcnt_hi((unsigned)(avail >> 32), __builtin_amdgcn_mbcnt_lo((unsigned)avail, 0u));
      const bool take = ((avail >> lane) & 1ull) && pos <= upto;
      if (take) lctl[4 + pos] = my_entry;
      const unsigned long long taken = __ballot(take);
      used |= taken;
      nsched += __builtin_popcountll(taken);
    };
    auto issue_remote = [&](unsigned e, int slot) {
      const int pq = (int)(e & 31u);
      LAS char* lwp = (LAS char*)smem + slot * SLOT + XB + lw * 32 * 128;
      const int sow = pq * 128 * N;
#pragma unroll
      for (int i = 0; i < 4; ++i)
        __builtin_amdgcn_raw_ptr_buffer_load_lds(rw, lwp + i * 1024, 16, (i & 2) ? vw1 : vw0, sow + i * 8 * N, 0, 0);
      LAS char* lx = (LAS char*)smem + slot * SLOT + lw * 64 * 128;
      const int sox = (pq >> 2) * BLK + (pq & 3) * 128;
#pragma unroll
      for (int i = 0; i < 8; ++i)
        __builtin_amdgcn_raw_ptr_buffer_load_lds(rxs, lx + i * 1024, 16, (i & 1) ? vx1 : vx0, sox + i * 8 * KS, 0, 17);
    };
    asm volatile("s_waitcnt vmcnt(12)" ::: "memory");
    BAR();
    issue_local(3, 0);
    int slot = 1;
    int t = 1;
    int poll_age = 0;
    unsigned tbl = 0;
    for (; t < NSTEP - 3; ++t) {
      if (poll_age == 1) asm volatile("s_waitcnt vmcnt(14)" ::: "memory");
      else asm volatile("s_waitcnt vmcnt(12)" ::: "memory");
      const int s = t + 3;
      if (lw == 0) {
        if (poll_age == 2) { ready |= poll_eval(); poll_age = 0; }
        schedule(s + 2);
      }
      unsigned e = __builtin_amdgcn_readlane(tbl, s);
      BAR();
      if (!e) {
        tbl = lctl[4 + (lane & 31)];
        e = __builtin_amdgcn_readlane(tbl, s);
        if (!e) {
          if (lw == 0) {
            const unsigned long long t0 = __builtin_amdgcn_s_memrealtime();
            for (unsigned n = 1;; ++n) {
              poll_issue();
              asm volatile("s_waitcnt vmcnt(0)" ::: "memory");
              ready |= poll_eval();
              if (ready & ~used) break;
              if ((n & 15u) == 0u) {
                const unsigned tv = __builtin_amdgcn_readfirstlane(__hip_atomic_load(tmo, __ATOMIC_RELAXED, __HIP_MEMORY_SCOPE_AGENT));
                if (tv == call || __builtin_amdgcn_s_memrealtime() - t0 > DEADLINE_TICKS) {
                  if (lane == 0) { __hip_atomic_store(tmo, call, __ATOMIC_RELAXED, __HIP_MEMORY_SCOPE_AGENT); lctl[0] = 1u; }
                  ready = ALL;
                  break;
                }
              }
            }
            poll_age = 0;
            schedule(s + 2);
          }
          for (;;) {
            tbl = lctl[4 + (lane & 31)];
            e = __builtin_amdgcn_readlane(tbl, s);
            if (e) break;
            __builtin_amdgcn_s_sleep(1);
          }
        }
      }
      const unsigned tbl_next = lctl[4 + (lane & 31)];
      issue_remote(e, slot);
      if (poll_age) poll_age = 2;
      else if (lw == 0 && ready != ALL) { poll_issue(); poll_age = 1; }
      tbl = tbl_next;
      slot = (slot == 2) ? 0 : slot + 1;
      if (__builtin_amdgcn_readlane(tbl, NSTEP - 1) != 0u && poll_age != 1) { ++t; break; }
    }
    for (; t < NSTEP - 3; ++t) {
      asm volatile("s_waitcnt vmcnt(12)" ::: "memory");
      const unsigned e = __builtin_amdgcn_readlane(tbl, t + 3);
      BAR();
      issue_remote(e, slot);
      slot = (slot == 2) ? 0 : slot + 1;
    }
    asm volatile("s_waitcnt vmcnt(12)" ::: "memory");
    BAR();
    asm volatile("s_waitcnt vmcnt(0)" ::: "memory");
    BAR();
    BAR();
    STAMP_FLUSH(12);
    return;
  }

  const bool pushes = (grp != rank);
  v4i px0, px1;
  const int poff = (idx * 16 + (lane >> 3)) * KS + wid * 128 + (lane & 7) * 16;
  char* const dws = P.ws[grp];
  unsigned pf_dummy = 0;
  const char* const pf_base = P.w + (size_t)(4 * idx + (lane >> 4)) * N + grp * 2048 + (lane & 15) * 128;
  auto pf_addr = [&](int s) {
    int p = rank, q = s;
    if (s >= 4) piece_pq(s - 4, rank, p, q);
    return pf_base + (size_t)(p * KS + q * 128) * N;
  };
  auto prefetch_w = [&](int s) {
    const char* a = pf_addr(s);
    asm volatile("global_load_dword %0, %1, off" : "+v"(pf_dummy) : "v"(a) : "memory");
  };
  unsigned call;
  {
    const char* src = P.x + (size_t)grp * BLK + poff;
    if (wid == 0) {
      unsigned* cp = (unsigned*)(myws + WS_CTR) + b * 32;
      unsigned tick = 0, one = 1u;
      asm volatile("s_mov_b64 exec, 1\n\tglobal_atomic_add %0, %3, %4, off sc0\n\ts_mov_b64 exec, -1\n\t"
                   "global_load_dwordx4 %1, %5, off\n\tglobal_load_dwordx4 %2, %6, off\n\ts_waitcnt vmcnt(0)"
                   : "+v"(tick), "=&v"(px0), "=&v"(px1) : "v"(cp), "v"(one), "v"(src), "v"(src + 8 * KS) : "memory");
      call = __builtin_amdgcn_readfirstlane(tick) + 1u;
      if (lane == 0) lctl[1] = call;
    } else {
      const int s0 = 3 + PF_START + (wid - 1);
      const char *a0 = pf_addr(s0), *a1 = pf_addr(s0 + 3), *a2 = pf_addr(s0 + 6), *a3 = pf_addr(s0 + 9);
      asm volatile("global_load_dwordx4 %0, %3, off\n\tglobal_load_dwordx4 %1, %4, off\n\t"
                   "s_waitcnt vmcnt(%9)"
                   : "=&v"(px0), "=&v"(px1), "+v"(pf_dummy) : "v"(src), "v"(src + 8 * KS), "v"(a0), "v"(a1), "v"(a2), "v"(a3), "n"(PF_PRE) : "memory");
      call = 0;
    }
  }
  STAMP(1);
  char* const pdst0 = dws + WS_X + (size_t)rank * BLK + poff;
  unsigned* const pflag = (unsigned*)(dws + WS_FLAG) + (((idx >> 4) * 8 + rank) * 4 + wid) * 16 + (idx & 15);
  auto push_stores = [&]() {
    char* const pdst = pdst0 + (size_t)(call & 1u) * NR * BLK;
    asm volatile("global_store_dwordx4 %0, %1, off sc0 sc1\n\ts_nop 1" :: "v"(pdst), "v"(px0) : "memory");
    asm volatile("global_store_dwordx4 %0, %1, off sc0 sc1\n\ts_nop 1" :: "v"(pdst + 8 * KS), "v"(px1) : "memory");
  };
  int pstate = pushes ? 0 : 2;
  const int t_store = T_STORE(wid);
  const int t_force = T_FORCE(wid);
  if (wid == 0 && pushes) { push_stores(); pstate = 1; }
  BAR();
  if (pushes && wid == 0) { asm volatile("s_waitcnt vmcnt(0)" ::: "memory"); if (lane == 0) __hip_atomic_store(pflag, call, __ATOMIC_RELAXED, __HIP_MEMORY_SCOPE_SYSTEM); pstate = 2; }
  int t_evt = pstate == 2 ? NSTEP : (pstate == 1 ? 0 : t_store);
  int t_mid = (pstate == 1) ? 0 : NSTEP;
  auto push_event = [&](int t, bool force) {
    if (pstate == 0) { push_stores(); pstate = 1; }
    else {
      unsigned sts; asm volatile("s_getreg_b32 %0, hwreg(HW_REG_IB_STS)" : "=s"(sts));
      if (force || (sts & 0x00C0000Fu) == 0u) {
        asm volatile("s_waitcnt vmcnt(0)" ::: "memory");
        if (lane == 0) __hip_atomic_store(pflag, call, __ATOMIC_RELAXED, __HIP_MEMORY_SCOPE_SYSTEM);
        pstate = 2; t_evt = NSTEP; t_mid = NSTEP;
      }
    }
  };

  const int wm = wid >> 1, wn = wid & 1;
  const int g = lane >> 4, l15 = lane & 15;
  const int fx = (l15 >> 1) & 7;
  const unsigned ax0 = (wm * 128 + l15) * 128 + ((g ^ fx) << 4);
  const unsigned ax1 = (wm * 128 + l15) * 128 + (((4 + g) ^ fx) << 4);
  const int q8 = l15 >> 1;
  unsigned aw[4];
#pragma unroll
  for (int j = 0; j < 4; ++j) aw[j] = XB + (16 * g + q8) * 128 + (((4 * wn + j) ^ (((q8 >> 1) & 3) | ((g & 1) << 2))) << 4) + 8 * (lane & 1);
  v4f acc[8][4];
#pragma unroll
  for (int i = 0; i < 8; ++i)
#pragma unroll
    for (int j = 0; j < 4; ++j) acc[i][j] = (v4f){};
  v8i alo[4], ahi[4], bf[4];
  const unsigned sb0 = (unsigned)(size_t)(LAS char*)smem;
#define RDA(dst, i, ii, sb) dst[i] = cat4(*(LAS v4i*)(size_t)(ax0 + (sb) + (ii) * 2048), *(LAS v4i*)(size_t)(ax1 + (sb) + (ii) * 2048))
#define TR(j, t, sb) __builtin_amdgcn_ds_read_tr8_b64_v2i32((LAS v2i*)(size_t)(aw[j] + (sb) + ((t) >> 1) * 8192 + ((t) & 1) * 1024))
#define RDB(j, sb) bf[j] = cat2(TR(j, 0, sb), TR(j, 1, sb), TR(j, 2, sb), TR(j, 3, sb))
#define MM(i, j, A, ai) acc[i][j] = __builtin_amdgcn_mfma_scale_f32_16x16x128_f8f6f4(A[ai], bf[j], acc[i][j], 1, 1, 0, 0, 0, 0)
#define MM4(i0, j, A) do { MM(i0, j, A, 0); MM(i0 + 1, j, A, 1); MM(i0 + 2, j, A, 2); MM(i0 + 3, j, A, 3); } while (0)
#define SGB(m, n) __builtin_amdgcn_sched_group_barrier(m, n, 0)
#define P1(n) SGB(0x8,1); SGB(0x100,n);
  WAIT_LGKM0();
  BAR();
  STAMP(2);
  call = __builtin_amdgcn_readfirstlane(lctl[1]);
  {
    const unsigned sb = sb0;
    RDA(alo, 0, 0, sb); RDA(alo, 1, 1, sb); RDA(alo, 2, 2, sb); RDA(alo, 3, 3, sb); RDB(0, sb); RDB(1, sb); RDB(2, sb);
  }
  unsigned sb = sb0;
  int slot = 0;
  const unsigned pf_voff = (4 * idx + (lane >> 4)) * N + grp * 2048 + (lane & 15) * 128;
  int pf_turn = 1, pf_q = (PF_D - 1) / 7, pf_i = (PF_D - 1) % 7;
  unsigned pf_em = PF_START == 0 ? (unsigned)(((1 ^ wid) - 1) >> 31) : 0u;
  const char* pf_ptr = P.w + (size_t)(unsigned)(((rank + 1 + pf_i) & 7) * KS + pf_q * 128) * N;
  for (int t = 0; t < NSTEP; ++t) {
    const int nslot = (slot == 2) ? 0 : slot + 1;
    const unsigned sbn = sb0 + nslot * SLOT;
    SCHED();
    RDB(3, sb);
    MM(0, 0, alo, 0);
    P1(4)
    SCHED();
    if (t >= t_evt) push_event(t, t >= t_force);
    SCHED();
    RDA(ahi, 0, 4, sb); RDA(ahi, 1, 5, sb); RDA(ahi, 2, 6, sb); RDA(ahi, 3, 7, sb);
    MM(1, 0, alo, 1); MM(2, 0, alo, 2); MM(3, 0, alo, 3);
    MM4(0, 1, alo); MM4(0, 2, alo); MM4(0, 3, alo);
    P1(1) P1(1) P1(1) P1(1) P1(1) P1(1) P1(1) P1(1) SGB(0x8, 7);
    SCHED();
    WAIT_LGKM0();
    BAR();
    SCHED();
    RDA(alo, 0, 0, sbn);
    MM(4, 0, ahi, 0);
    P1(2)
    SCHED();
    if (t >= t_mid) push_event(t, false);
    SCHED();
    RDA(alo, 1, 1, sbn); RDA(alo, 2, 2, sbn); RDA(alo, 3, 3, sbn);
    MM(5, 0, ahi, 1); MM(6, 0, ahi, 2); MM(7, 0, ahi, 3);
    P1(2) P1(2) P1(2)
    SCHED();
    RDB(0, sbn);
    MM4(4, 1, ahi);
    P1(1) P1(1) P1(1) P1(1)
    SCHED();
    RDB(1, sbn);
    MM4(4, 2, ahi);
    P1(1) P1(1) P1(1) P1(1)
    SCHED();
    RDB(2, sbn);
    MM4(4, 3, ahi);
    P1(1) P1(1) P1(1) P1(1)
    SCHED();
    sb = sbn; slot = nslot;
  }
  STAMP(6);
  asm volatile("s_waitcnt vmcnt(0)" : "+v"(pf_dummy) :: "memory");
  const unsigned tmo_seen = __hip_atomic_load(tmo, __ATOMIC_RELAXED, __HIP_MEMORY_SCOPE_AGENT);
  const bool bad = (tmo_seen == call) || (lctl[0] != 0u);
  const float s = bad ? __builtin_nanf("") : P.sx[0] * P.sw[0];
  __amdgpu_buffer_rsrc_t ro = __builtin_amdgcn_make_buffer_rsrc((void*)P.out, 0, MS * N * 4, 0x00020000);
  const int vo = ((m0 + wm * 128 + 8 * (lane >> 5)) * N + n0 + wn * 64 + (lane & 31)) * 4;
#pragma unroll
  for (int i = 0; i < 8; ++i)
#pragma unroll
    for (int r = 0; r < 4; ++r)
#pragma unroll
      for (int jp = 0; jp < 2; ++jp) {
        unsigned o[2];
#pragma unroll
        for (int jj = 0; jj < 2; ++jj) {
          const float y = acc[i][2 * jp + jj][r] * s;
          const float c = __builtin_amdgcn_fmed3f(y, -60.f, 60.f);
          const float e = __builtin_amdgcn_exp2f(c * -1.44269504088896f);
          o[jj] = __builtin_bit_cast(unsigned, y * __builtin_amdgcn_rcpf(1.0f + e));
        }
        const auto sw2 = __builtin_amdgcn_permlane16_swap(o[0], o[1], false, false);
        __builtin_amdgcn_raw_buffer_store_b32(sw2[0], ro, vo + jp * 128, (16 * i + r) * N * 4, 16);
        __builtin_amdgcn_raw_buffer_store_b32(sw2[1], ro, vo + jp * 128, (16 * i + r + 4) * N * 4, 16);
      }
}

extern "C" void kernel_launch(int rank, int nranks, ncclComm_t comm,
                              hipStream_t stream,
                              void* const* d_in, const int* in_sizes, int n_in,
                              void* d_out, int out_size,
                              const int* in_dtypes, int out_dtype,
                              void* const* d_out_all,
                              void* const* d_ws_all, size_t ws_bytes) {
  (void)comm; (void)in_dtypes; (void)out_dtype; (void)d_out_all;
  if (nranks != NR || rank < 0 || rank >= NR || n_in != 4 || in_sizes[0] != NR * BLK || in_sizes[1] != K * N ||
      out_size != MS * N || ws_bytes < WS_NEED) {
    fprintf(stderr, "kernel_launch: unexpected problem shape\n");
    return;
  }
  static bool attr_set = false;
  if (!attr_set) {
    HIP_CHECK(hipFuncSetAttribute((const void*)a2a_gemm_k, hipFuncAttributeMaxDynamicSharedMemorySize, LDS_BYTES));
    attr_set = true;
  }
  Params P;
  P.x = (const char*)d_in[0]; P.w = (const char*)d_in[1]; P.sx = (const float*)d_in[2]; P.sw = (const float*)d_in[3];
  P.out = (float*)d_out; P.myws = (char*)d_ws_all[rank];
  for (int p = 0; p < NR; ++p) P.ws[p] = (char*)d_ws_all[p];
  P.rank = rank; P.pad = 0;
  a2a_gemm_k<<<256, 512, LDS_BYTES, stream>>>(P);
  HIP_CHECK(hipGetLastError());
}
```

```cpp
#include <hip/hip_runtime.h>
#include <rccl/rccl.h>
#include <cstdio>

#define HIP_CHECK(expr) do { hipError_t _e = (expr); if (_e != hipSuccess) { \
    fprintf(stderr, "HIP error %s at %s:%d\n", hipGetErrorString(_e), __FILE__, __LINE__); \
    return; } } while (0)

#define LAS __attribute__((address_space(3)))
typedef int v2i __attribute__((ext_vector_type(2)));
typedef int v4i __attribute__((ext_vector_type(4)));
typedef int v8i __attribute__((ext_vector_type(8)));
typedef unsigned v4u __attribute__((ext_vector_type(4)));
typedef float v4f __attribute__((ext_vector_type(4)));

constexpr int NR = 8;
constexpr int MS = 512, N = 16384, K = 4096;
constexpr int KS = K / NR;
constexpr int BLK = MS * KS;
constexpr int SLOT = 49152, XB = 32768;
constexpr int NSLOT = 3, NSTEP = 32, NREM = 28;
constexpr int LDS_CTRL = NSLOT * SLOT;
constexpr int LDS_FLG = LDS_CTRL + 256;
constexpr int LDS_BYTES = LDS_FLG + 2048 + 256;
constexpr size_t WS_CTR = 0;
constexpr size_t WS_TMO = 32768;
constexpr size_t WS_FLAG = 36864;
constexpr size_t WS_X = 1 << 20;
constexpr size_t WS_NEED = WS_X + 2 * (size_t)NR * BLK;
constexpr unsigned long long DEADLINE_TICKS = 100000000ull;

#define POLL_LOW 5
#define POLL_EVERY 3
#define POLL_URGENT 3
#define FIRST_LOOK 0
#define T_STORE(w, j) ((w) == 0 ? -1 : 2 * ((w) - 1))
#define PF_DIST 0
#define PF_PRE 0
#define PF_START 0
constexpr int PF_D = PF_DIST;

struct Params {
  const char* x; const char* w; const float* sx; const float* sw; float* out; char* myws;
  char* ws[NR];
  int rank; int pad;
};

#define LTRACE(i) do {} while (0)
#define STAMP(i) do {} while (0)
#define STAMP_FLUSH(n) do {} while (0)

__device__ __forceinline__ v8i cat4(v4i lo, v4i hi) { return __builtin_shufflevector(lo, hi, 0, 1, 2, 3, 4, 5, 6, 7); }
__device__ __forceinline__ v8i cat2(v2i a, v2i b, v2i c, v2i d) {
  v4i lo = __builtin_shufflevector(a, b, 0, 1, 2, 3), hi = __builtin_shufflevector(c, d, 0, 1, 2, 3);
  return cat4(lo, hi);
}
#define WAIT_LGKM0() __builtin_amdgcn_s_waitcnt(0xC07F)
#define BAR() __builtin_amdgcn_s_barrier()
#define SCHED() __builtin_amdgcn_sched_barrier(0)

__device__ __forceinline__ void piece_pq(int u, int rank, int& p, int& q) { q = u / 7; p = (rank + 1 + (u - q * 7)) & 7; }

__global__ __launch_bounds__(512, 2) void a2a_gemm_k(const Params P) {
  extern __shared__ __attribute__((aligned(16))) char smem[];
  const int tid = threadIdx.x, wid = __builtin_amdgcn_readfirstlane(tid >> 6), lane = tid & 63;
  const int b = blockIdx.x, grp = b & 7, idx = b >> 3;
  const int mh = idx & 1, nt = grp * 16 + (idx >> 1);
  const int m0 = mh * 256, n0 = nt * 128;
  const int rank = P.rank;
  char* const myws = P.myws;
  STAMP(0);
  unsigned* const tmo = (unsigned*)(myws + WS_TMO);
  volatile LAS unsigned* const lctl = (volatile LAS unsigned*)((LAS char*)smem + LDS_CTRL);

  if (wid >= 4) {
    const int lw = wid - 4;
    __amdgpu_buffer_rsrc_t rxl = __builtin_amdgcn_make_buffer_rsrc((void*)(P.x + (size_t)rank * BLK), 0, BLK, 0x00020000);
    __amdgpu_buffer_rsrc_t rw = __builtin_amdgcn_make_buffer_rsrc((void*)P.w, 0, K * N, 0x00020000);
    const int r8 = lane >> 3, sl = lane & 7;
    const int vx0 = (m0 + lw * 64 + r8) * KS + ((sl ^ (r8 >> 1)) << 4);
    const int vx1 = (m0 + lw * 64 + r8) * KS + ((sl ^ (4 + (r8 >> 1))) << 4);
    const int vs0 = (m0 + lw * 64 + r8) * 128 + ((sl ^ (r8 >> 1)) << 4);
    const int vs1 = (m0 + lw * 64 + r8) * 128 + ((sl ^ (4 + (r8 >> 1))) << 4);
    const int vw0 = (lw * 32 + r8) * N + n0 + ((sl ^ ((r8 >> 1) & 3)) << 4);
    const int vw1 = (lw * 32 + r8) * N + n0 + ((sl ^ (((r8 >> 1) & 3) | 4)) << 4);
    auto issue_w = [&](int p, int q, int slot) {
      const int sow = (p * KS + q * 128) * N;
      LAS char* lwp = (LAS char*)smem + slot * SLOT + XB + lw * 32 * 128;
#pragma unroll
      for (int i = 0; i < 4; ++i)
        __builtin_amdgcn_raw_ptr_buffer_load_lds(rw, lwp + i * 1024, 16, (i & 2) ? vw1 : vw0, sow + i * 8 * N, 0, 0);
    };
    auto issue_local = [&](int s, int slot) {
      issue_w(rank, s, slot);
      LAS char* lx = (LAS char*)smem + slot * SLOT + lw * 64 * 128;
#pragma unroll
      for (int i = 0; i < 8; ++i)
        __builtin_amdgcn_raw_ptr_buffer_load_lds(rxl, lx + i * 1024, 16, (i & 1) ? vx1 : vx0, s * 128 + i * 8 * KS, 0, 0);
    };
    issue_local(0, 0); issue_local(1, 1);
    if (lw == 0 && lane < 40 && lane != 1) lctl[lane] = 0u;
    asm volatile("s_waitcnt vmcnt(12) lgkmcnt(0)" ::: "memory");
    BAR();
    issue_local(2, 2);
    LTRACE(32);
    STAMP(1);
    const unsigned call = __builtin_amdgcn_readfirstlane(lctl[1]);
    __amdgpu_buffer_rsrc_t rxs = __builtin_amdgcn_make_buffer_rsrc((void*)(myws + WS_X + (size_t)(call & 1u) * NR * BLK), 0, NR * BLK, 0x00020000);
    __amdgpu_buffer_rsrc_t rfl = __builtin_amdgcn_make_buffer_rsrc((void*)(myws + WS_FLAG + mh * 2048), 0, 2048, 0x00020000);
    unsigned flg_addr, my_entry;
    {
      int p, q; piece_pq(min(lane >> 1, NREM - 1), rank, p, q);
      flg_addr = (unsigned)(size_t)((LAS char*)smem + LDS_FLG) + (p * 4 + q) * 64 + (lane & 1) * 32;
      my_entry = 0x100u | (unsigned)(p * 4 + q);
    }
    auto poll_issue = [&]() {
      __builtin_amdgcn_raw_ptr_buffer_load_lds(rfl, (LAS char*)smem + LDS_FLG, 16, lane * 16, 0, 0, 17);
      __builtin_amdgcn_raw_ptr_buffer_load_lds(rfl, (LAS char*)smem + LDS_FLG + 1024, 16, lane * 16, 1024, 0, 17);
    };
    auto poll_eval = [&]() -> unsigned long long {
      const v4u f0 = *(volatile LAS v4u*)(size_t)flg_addr, f1 = *(volatile LAS v4u*)(size_t)(flg_addr + 16);
      const unsigned m = min(min(min(f0.x, f0.y), min(f0.z, f0.w)), min(min(f1.x, f1.y), min(f1.z, f1.w)));
      const unsigned long long ok = __ballot(m >= call && lane < 2 * NREM);
      return ok & (ok >> 1) & 0x5555555555555555ull;
    };
    unsigned long long ready = 0, used = 0;
    constexpr unsigned long long ALL = 0x0055555555555555ull;
    int nsched = 4;
    auto schedule = [&](int upto) {
      const unsigned long long avail = ready & ~used;
      const int q0 = (__builtin_ctzll(ALL & ~used) >> 1) / 7;
      const unsigned long long gq = 0x1555ull << (14 * q0);
      const unsigned long long first = ready == ALL ? avail : (avail & gq), rest = avail & ~first;
      const int nfirst = __builtin_popcountll(first);
      const bool in_first = (first >> lane) & 1ull;
      const unsigned long long m = in_first ? first : rest;
      const int pos = nsched + (in_first ? 0 : nfirst) + (int)__builtin_amdgcn_mbcnt_hi((unsigned)(m >> 32), __builtin_amdgcn_mbcnt_lo((unsigned)m, 0u));
      const bool take = ((avail >> lane) & 1ull) && (in_first || pos <= upto);
      if (take) lctl[4 + pos] = my_entry;
      const unsigned long long taken = __ballot(take);
      used |= taken;
      nsched += __builtin_popcountll(taken);
    };
    auto issue_remote = [&](unsigned e, int slot) {
      const int pq = (int)(e & 31u);
      LAS char* lwp = (LAS char*)smem + slot * SLOT + XB + lw * 32 * 128;
      const int sow = pq * 128 * N;
#pragma unroll
      for (int i = 0; i < 4; ++i)
        __builtin_amdgcn_raw_ptr_buffer_load_lds(rw, lwp + i * 1024, 16, (i & 2) ? vw1 : vw0, sow + i * 8 * N, 0, 0);
      LAS char* lx = (LAS char*)smem + slot * SLOT + lw * 64 * 128;
      const int sox = pq * (MS * 128);
#pragma unroll
      for (int i = 0; i < 8; ++i)
        __builtin_amdgcn_raw_ptr_buffer_load_lds(rxs, lx + i * 1024, 16, (i & 1) ? vs1 : vs0, sox + i * 1024, 0, 17);
    };
    asm volatile("s_waitcnt vmcnt(12)" ::: "memory");
    BAR();
    LTRACE(0);
    int poll_age = 0;
    if (FIRST_LOOK && lw == 0 && ready != ALL) { poll_issue(); poll_age = 3; }
    issue_local(3, 0);
    int slot = 1;
    int t = 1;
    unsigned tbl = 0;
    for (; t < NSTEP - 3; ++t) {
      if (poll_age == 1 || poll_age == 2) asm volatile("s_waitcnt vmcnt(14)" ::: "memory");
      else asm volatile("s_waitcnt vmcnt(12)" ::: "memory");
      const int s = t + 3;
      if (lw == 0) {
        if (poll_age == 2) {
          unsigned sts; asm volatile("s_getreg_b32 %0, hwreg(HW_REG_IB_STS)" : "=s"(sts));
          if (((sts & 15u) | ((sts >> 18) & 0x30u)) <= 12u) { asm volatile("s_waitcnt vmcnt(12)" ::: "memory"); poll_age = 3; }
        }
        bool news = false;
        if (poll_age == 3) { ready |= poll_eval(); poll_age = 0; news = true; }
        if ((news || nsched <= s + 2) && (ready & ~used)) schedule(s + 2);
      }
      unsigned e = __builtin_amdgcn_readlane(tbl, s);
      BAR();
      LTRACE(t);
      if (!e) {
        tbl = lctl[4 + (lane & 31)];
        e = __builtin_amdgcn_readlane(tbl, s);
        if (!e) {
          if (lw == 0) {
            const unsigned long long t0 = __builtin_amdgcn_s_memrealtime();
            for (unsigned n = 1;; ++n) {
              poll_issue();
              asm volatile("s_waitcnt vmcnt(0)" ::: "memory");
              ready |= poll_eval();
              if (ready & ~used) break;
              if ((n & 15u) == 0u) {
                const unsigned tv = __builtin_amdgcn_readfirstlane(__hip_atomic_load(tmo, __ATOMIC_RELAXED, __HIP_MEMORY_SCOPE_AGENT));
                if (tv == call || __builtin_amdgcn_s_memrealtime() - t0 > DEADLINE_TICKS) {
                  if (lane == 0) { __hip_atomic_store(tmo, call, __ATOMIC_RELAXED, __HIP_MEMORY_SCOPE_AGENT); lctl[0] = 1u; }
                  ready = ALL;
                  break;
                }
              }
            }
            poll_age = 0;
            schedule(s + 2);
          }
          for (;;) {
            tbl = lctl[4 + (lane & 31)];
            e = __builtin_amdgcn_readlane(tbl, s);
            if (e) break;
            __builtin_amdgcn_s_sleep(1);
          }
        }
      }
      const unsigned tbl_next = lctl[4 + (lane & 31)];
      const int have = (lw == 0 && ready != ALL && poll_age == 0) ? nsched - s - 1 + __builtin_popcountll(ready & ~used) : 99;
      if (have < POLL_URGENT) poll_issue();
      issue_remote(e, slot);
      if (poll_age) ++poll_age;
      else if (have < POLL_URGENT) poll_age = 3;
      else if (have < POLL_LOW || (have < 99 && (t & POLL_EVERY) == 0)) { poll_issue(); poll_age = 1; }
      tbl = tbl_next;
      slot = (slot == 2) ? 0 : slot + 1;
      if (__builtin_amdgcn_readlane(tbl, NSTEP - 1) != 0u && poll_age == 0) { ++t; break; }
    }
    for (; t < NSTEP - 3; ++t) {
      asm volatile("s_waitcnt vmcnt(12)" ::: "memory");
      const unsigned e = __builtin_amdgcn_readlane(tbl, t + 3);
      BAR();
      LTRACE(t);
      issue_remote(e, slot);
      slot = (slot == 2) ? 0 : slot + 1;
    }
    asm volatile("s_waitcnt vmcnt(12)" ::: "memory");
    BAR();
    LTRACE(29);
    asm volatile("s_waitcnt vmcnt(0)" ::: "memory");
    BAR();
    LTRACE(30);
    BAR();
    LTRACE(31);
    STAMP_FLUSH(12);
    return;
  }

  const int dst = (grp + idx) & 7;
  const bool pushes = (dst != rank);
  v4i px0, px1;
  const int poff = (idx * 16 + (lane >> 3)) * KS + wid * 128 + (lane & 7) * 16;
  char* const dws = P.ws[dst];
  unsigned pf_dummy = 0;
  const char* const pf_base = P.w + (size_t)(4 * idx + (lane >> 4)) * N + grp * 2048 + (lane & 15) * 128;
  auto pf_addr = [&](int s) {
    int p = rank, q = s;
    if (s >= 4) piece_pq(s - 4, rank, p, q);
    return pf_base + (size_t)(p * KS + q * 128) * N;
  };
  auto prefetch_w = [&](int s) {
    const char* a = pf_addr(s);
    asm volatile("global_load_dword %0, %1, off" : "+v"(pf_dummy) : "v"(a) : "memory");
  };
  unsigned call;
  {
    const char* src = P.x + (size_t)dst * BLK + poff;
    if (wid == 0) {
      unsigned* cp = (unsigned*)(myws + WS_CTR) + b * 32;
      unsigned tick = 0, one = 1u;
      asm volatile("s_mov_b64 exec, 1\n\tglobal_atomic_add %0, %3, %4, off sc0\n\ts_mov_b64 exec, -1\n\t"
                   "global_load_dwordx4 %1, %5, off\n\tglobal_load_dwordx4 %2, %6, off\n\ts_waitcnt vmcnt(0)"
                   : "+v"(tick), "=&v"(px0), "=&v"(px1) : "v"(cp), "v"(one), "v"(src), "v"(src + 8 * KS) : "memory");
      call = __builtin_amdgcn_readfirstlane(tick) + 1u;
      if (lane == 0) lctl[1] = call;
    } else {
      const int s0 = 3 + PF_START + (wid - 1);
      const char *a0 = pf_addr(s0), *a1 = pf_addr(s0 + 3), *a2 = pf_addr(s0 + 6), *a3 = pf_addr(s0 + 9);
      asm volatile("global_load_dwordx4 %0, %3, off\n\tglobal_load_dwordx4 %1, %4, off\n\t"
                   "s_waitcnt vmcnt(%9)"
                   : "=&v"(px0), "=&v"(px1), "+v"(pf_dummy) : "v"(src), "v"(src + 8 * KS), "v"(a0), "v"(a1), "v"(a2), "v"(a3), "n"(PF_PRE) : "memory");
      call = 0;
    }
  }
  STAMP(1);
  char* const pdst0 = dws + WS_X + (size_t)rank * BLK + wid * (MS * 128) + (idx * 16 + (lane >> 3)) * 128 + (lane & 7) * 16;
  unsigned* const pflag = (unsigned*)(dws + WS_FLAG) + (((idx >> 4) * 8 + rank) * 4 + wid) * 16 + (idx & 15);
  auto push_stores = [&]() {
    char* const pdst = pdst0 + (size_t)(call & 1u) * NR * BLK;
    asm volatile("global_store_dwordx4 %0, %1, off sc0 sc1\n\ts_nop 1" :: "v"(pdst), "v"(px0) : "memory");
    asm volatile("global_store_dwordx4 %0, %1, off offset:1024 sc0 sc1\n\ts_nop 1" :: "v"(pdst), "v"(px1) : "memory");
  };
  const int pj = (rank - 1 - dst) & 7;
  const int t_store = T_STORE(wid, pj);
  const int t_force = (t_store < 0 ? -1 : t_store) + 3;
  int pstate = pushes ? 0 : 2;
  if (pushes && t_store < 0) { push_stores(); pstate = 1; }
  STAMP(10);
  if (pstate == 1) { asm volatile("s_waitcnt vmcnt(0)" ::: "memory"); STAMP(11); if (lane == 0) __hip_atomic_store(pflag, call, __ATOMIC_RELAXED, __HIP_MEMORY_SCOPE_SYSTEM); pstate = 2; }
  int t_evt = pstate == 2 ? NSTEP : (pstate == 1 ? 0 : t_store);
  int t_mid = pstate == 1 ? 0 : NSTEP;
  auto push_event = [&](int t, bool force) {
    if (pstate == 0) { push_stores(); pstate = 1; if (wid == 0) t_mid = t; }
    else {
      unsigned sts; asm volatile("s_getreg_b32 %0, hwreg(HW_REG_IB_STS)" : "=s"(sts));
      if (force || (sts & 0x00C0000Fu) == 0u) {
        asm volatile("s_waitcnt vmcnt(0)" ::: "memory");
        if (lane == 0) __hip_atomic_store(pflag, call, __ATOMIC_RELAXED, __HIP_MEMORY_SCOPE_SYSTEM);
        pstate = 2; t_evt = NSTEP; t_mid = NSTEP;
      }
    }
  };

  const int wm = wid >> 1, wn = wid & 1;
  const int g = lane >> 4, l15 = lane & 15;
  const int fx = (l15 >> 1) & 7;
  const unsigned ax0 = (wm * 128 + l15) * 128 + ((g ^ fx) << 4);
  const unsigned ax1 = (wm * 128 + l15) * 128 + (((4 + g) ^ fx) << 4);
  const int q8 = l15 >> 1;
  unsigned aw[4];
#pragma unroll
  for (int j = 0; j < 4; ++j) aw[j] = XB + (16 * g + q8) * 128 + (((4 * wn + j) ^ (((q8 >> 1) & 3) | ((g & 1) << 2))) << 4) + 8 * (lane & 1);
  v4f acc[8][4];
#pragma unroll
  for (int i = 0; i < 8; ++i)
#pragma unroll
    for (int j = 0; j < 4; ++j) acc[i][j] = (v4f){};
  v8i alo[4], ahi[4], bf[4];
  const unsigned sb0 = (unsigned)(size_t)(LAS char*)smem;
#define RDA(dst, i, ii, sb) dst[i] = cat4(*(LAS v4i*)(size_t)(ax0 + (sb) + (ii) * 2048), *(LAS v4i*)(size_t)(ax1 + (sb) + (ii) * 2048))
#define TR(j, t, sb) __builtin_amdgcn_ds_read_tr8_b64_v2i32((LAS v2i*)(size_t)(aw[j] + (sb) + ((t) >> 1) * 8192 + ((t) & 1) * 1024))
#define RDB(j, sb) bf[j] = cat2(TR(j, 0, sb), TR(j, 1, sb), TR(j, 2, sb), TR(j, 3, sb))
#define MM(i, j, A, ai) acc[i][j] = __builtin_amdgcn_mfma_scale_f32_16x16x128_f8f6f4(A[ai], bf[j], acc[i][j], 1, 1, 0, 0, 0, 0)
#define MM4(i0, j, A) do { MM(i0, j, A, 0); MM(i0 + 1, j, A, 1); MM(i0 + 2, j, A, 2); MM(i0 + 3, j, A, 3); } while (0)
#define SGB(m, n) __builtin_amdgcn_sched_group_barrier(m, n, 0)
#define P1(n) SGB(0x8,1); SGB(0x100,n);
  WAIT_LGKM0();
  BAR();
  STAMP(2);
  call = __builtin_amdgcn_readfirstlane(lctl[1]);
  {
    const unsigned sb = sb0;
    RDA(alo, 0, 0, sb); RDA(alo, 1, 1, sb); RDA(alo, 2, 2, sb); RDA(alo, 3, 3, sb); RDB(0, sb); RDB(1, sb); RDB(2, sb);
  }
  unsigned sb = sb0;
  int slot = 0;
  const unsigned pf_voff = (4 * idx + (lane >> 4)) * N + grp * 2048 + (lane & 15) * 128;
  int pf_turn = 1, pf_q = (PF_D - 1) / 7, pf_i = (PF_D - 1) % 7;
  unsigned pf_em = PF_START == 0 ? (unsigned)(((1 ^ wid) - 1) >> 31) : 0u;
  const char* pf_ptr = P.w + (size_t)(unsigned)(((rank + 1 + pf_i) & 7) * KS + pf_q * 128) * N;
  for (int t = 0; t < NSTEP; ++t) {
    const int nslot = (slot == 2) ? 0 : slot + 1;
    const unsigned sbn = sb0 + nslot * SLOT;
    SCHED();
    RDB(3, sb);
    MM(0, 0, alo, 0);
    P1(4)
    SCHED();
    if (t >= t_evt) push_event(t, t >= t_force);
    SCHED();
    RDA(ahi, 0, 4, sb); RDA(ahi, 1, 5, sb); RDA(ahi, 2, 6, sb); RDA(ahi, 3, 7, sb);
    MM(1, 0, alo, 1); MM(2, 0, alo, 2); MM(3, 0, alo, 3);
    MM4(0, 1, alo); MM4(0, 2, alo); MM4(0, 3, alo);
    P1(1) P1(1) P1(1) P1(1) P1(1) P1(1) P1(1) P1(1) SGB(0x8, 7);
    SCHED();
    WAIT_LGKM0();
    BAR();
    SCHED();
    RDA(alo, 0, 0, sbn);
    MM(4, 0, ahi, 0);
    P1(2)
    SCHED();
    if (t >= t_mid) push_event(t, false);
    SCHED();
    RDA(alo, 1, 1, sbn); RDA(alo, 2, 2, sbn); RDA(alo, 3, 3, sbn);
    MM(5, 0, ahi, 1); MM(6, 0, ahi, 2); MM(7, 0, ahi, 3);
    P1(2) P1(2) P1(2)
    SCHED();
    RDB(0, sbn);
    MM4(4, 1, ahi);
    P1(1) P1(1) P1(1) P1(1)
    SCHED();
    RDB(1, sbn);
    MM4(4, 2, ahi);
    P1(1) P1(1) P1(1) P1(1)
    SCHED();
    RDB(2, sbn);
    MM4(4, 3, ahi);
    P1(1) P1(1) P1(1) P1(1)
    SCHED();
    sb = sbn; slot = nslot;
  }
  STAMP(6);
  asm volatile("s_waitcnt vmcnt(0)" : "+v"(pf_dummy) :: "memory");
  const unsigned tmo_seen = __hip_atomic_load(tmo, __ATOMIC_RELAXED, __HIP_MEMORY_SCOPE_AGENT);
  const bool bad = (tmo_seen == call) || (lctl[0] != 0u);
  const float s = bad ? __builtin_nanf("") : P.sx[0] * P.sw[0];
  __amdgpu_buffer_rsrc_t ro = __builtin_amdgcn_make_buffer_rsrc((void*)P.out, 0, MS * N * 4, 0x00020000);
  const int vo = ((m0 + wm * 128 + 8 * (lane >> 5)) * N + n0 + wn * 64 + (lane & 31)) * 4;
#pragma unroll
  for (int i = 0; i < 8; ++i)
#pragma unroll
    for (int r = 0; r < 4; ++r)
#pragma unroll
      for (int jp = 0; jp < 2; ++jp) {
        unsigned o[2];
#pragma unroll
        for (int jj = 0; jj < 2; ++jj) {
          const float y = acc[i][2 * jp + jj][r] * s;
          const float c = __builtin_amdgcn_fmed3f(y, -60.f, 60.f);
          const float e = __builtin_amdgcn_exp2f(c * -1.44269504088896f);
          o[jj] = __builtin_bit_cast(unsigned, y * __builtin_amdgcn_rcpf(1.0f + e));
        }
        const auto sw2 = __builtin_amdgcn_permlane16_swap(o[0], o[1], false, false);
        __builtin_amdgcn_raw_buffer_store_b32(sw2[0], ro, vo + jp * 128, (16 * i + r) * N * 4, 16);
        __builtin_amdgcn_raw_buffer_store_b32(sw2[1], ro, vo + jp * 128, (16 * i + r + 4) * N * 4, 16);
      }
}

extern "C" void kernel_launch(int rank, int nranks, ncclComm_t comm,
                              hipStream_t stream,
                              void* const* d_in, const int* in_sizes, int n_in,
                              void* d_out, int out_size,
                              const int* in_dtypes, int out_dtype,
                              void* const* d_out_all,
                              void* const* d_ws_all, size_t ws_bytes) {
  (void)comm; (void)in_dtypes; (void)out_dtype; (void)d_out_all;
  if (nranks != NR || rank < 0 || rank >= NR || n_in != 4 || in_sizes[0] != NR * BLK || in_sizes[1] != K * N ||
      out_size != MS * N || ws_bytes < WS_NEED) {
    fprintf(stderr, "kernel_launch: unexpected problem shape\n");
    return;
  }
  static bool attr_set = false;
  if (!attr_set) {
    HIP_CHECK(hipFuncSetAttribute((const void*)a2a_gemm_k, hipFuncAttributeMaxDynamicSharedMemorySize, LDS_BYTES));
    attr_set = true;
  }
  Params P;
  P.x = (const char*)d_in[0]; P.w = (const char*)d_in[1]; P.sx = (const float*)d_in[2]; P.sw = (const float*)d_in[3];
  P.out = (float*)d_out; P.myws = (char*)d_ws_all[rank];
  for (int p = 0; p < NR; ++p) P.ws[p] = (char*)d_ws_all[p];
  P.rank = rank; P.pad = 0;
  a2a_gemm_k<<<256, 512, LDS_BYTES, stream>>>(P);
  HIP_CHECK(hipGetLastError());
}
```

```cpp
#include <hip/hip_runtime.h>
#include <rccl/rccl.h>
#include <cstdio>

#define HIP_CHECK(expr) do { hipError_t _e = (expr); if (_e != hipSuccess) { \
    fprintf(stderr, "HIP error %s at %s:%d\n", hipGetErrorString(_e), __FILE__, __LINE__); \
    return; } } while (0)

#define LAS __attribute__((address_space(3)))
typedef int v2i __attribute__((ext_vector_type(2)));
typedef int v4i __attribute__((ext_vector_type(4)));
typedef int v8i __attribute__((ext_vector_type(8)));
typedef unsigned v4u __attribute__((ext_vector_type(4)));
typedef float v4f __attribute__((ext_vector_type(4)));

constexpr int NR = 8;
constexpr int MS = 512, N = 16384, K = 4096;
constexpr int KS = K / NR;
constexpr int BLK = MS * KS;
constexpr int SLOT = 49152, XB = 32768;
constexpr int NSLOT = 3, NSTEP = 32, NREM = 28;
constexpr int LDS_CTRL = NSLOT * SLOT;
constexpr int LDS_FLG = LDS_CTRL + 256;
constexpr int LDS_BYTES = LDS_FLG + 2048 + 256;
constexpr size_t WS_CTR = 0;
constexpr size_t WS_TMO = 32768;
constexpr size_t WS_FLAG = 36864;
constexpr size_t WS_X = 1 << 20;
constexpr size_t WS_NEED = WS_X + 2 * (size_t)NR * BLK;
constexpr unsigned long long DEADLINE_TICKS = 100000000ull;

#define POLL_LOW 5
#define POLL_EVERY 3
#define POLL_URGENT 3
#define FIRST_LOOK 0
#define T_STORE(w, j) ((w) == 0 ? -1 : 2 * ((w) - 1))
#define PF_DIST 5
#define PF_START 5
constexpr int PF_D = PF_DIST;

struct Params {
  const char* x; const char* w; const float* sx; const float* sw; float* out; char* myws;
  char* ws[NR];
  int rank; int pad;
};

#define LTRACE(i) do {} while (0)
#define STAMP(i) do {} while (0)
#define STAMP_FLUSH(n) do {} while (0)

__device__ __forceinline__ v8i cat4(v4i lo, v4i hi) { return __builtin_shufflevector(lo, hi, 0, 1, 2, 3, 4, 5, 6, 7); }
__device__ __forceinline__ v8i cat2(v2i a, v2i b, v2i c, v2i d) {
  v4i lo = __builtin_shufflevector(a, b, 0, 1, 2, 3), hi = __builtin_shufflevector(c, d, 0, 1, 2, 3);
  return cat4(lo, hi);
}
#define WAIT_LGKM0() __builtin_amdgcn_s_waitcnt(0xC07F)
#define BAR() __builtin_amdgcn_s_barrier()
#define SCHED() __builtin_amdgcn_sched_barrier(0)

__device__ __forceinline__ void piece_pq(int u, int rank, int& p, int& q) { q = u / 7; p = (rank + 1 + (u - q * 7)) & 7; }

__global__ __launch_bounds__(512, 2) void a2a_gemm_k(const Params P) {
  extern __shared__ __attribute__((aligned(16))) char smem[];
  const int tid = threadIdx.x, wid = __builtin_amdgcn_readfirstlane(tid >> 6), lane = tid & 63;
  const int b = blockIdx.x, grp = b & 7, idx = b >> 3;
  const int mh = idx & 1, nt = grp * 16 + (idx >> 1);
  const int m0 = mh * 256, n0 = nt * 128;
  const int rank = P.rank;
  char* const myws = P.myws;
  STAMP(0);
  unsigned* const tmo = (unsigned*)(myws + WS_TMO);
  volatile LAS unsigned* const lctl = (volatile LAS unsigned*)((LAS char*)smem + LDS_CTRL);

  if (wid >= 4) {
    const int lw = wid - 4;
    __amdgpu_buffer_rsrc_t rxl = __builtin_amdgcn_make_buffer_rsrc((void*)(P.x + (size_t)rank * BLK), 0, BLK, 0x00020000);
    __amdgpu_buffer_rsrc_t rw = __builtin_amdgcn_make_buffer_rsrc((void*)P.w, 0, K * N, 0x00020000);
    const int r8 = lane >> 3, sl = lane & 7;
    const int vx0 = (m0 + lw * 64 + r8) * KS + ((sl ^ (r8 >> 1)) << 4);
    const int vx1 = (m0 + lw * 64 + r8) * KS + ((sl ^ (4 + (r8 >> 1))) << 4);
    const int vs0 = (m0 + lw * 64 + r8) * 128 + ((sl ^ (r8 >> 1)) << 4);
    const int vs1 = (m0 + lw * 64 + r8) * 128 + ((sl ^ (4 + (r8 >> 1))) << 4);
    const int vw0 = (lw * 32 + r8) * N + n0 + ((sl ^ ((r8 >> 1) & 3)) << 4);
    const int vw1 = (lw * 32 + r8) * N + n0 + ((sl ^ (((r8 >> 1) & 3) | 4)) << 4);
    auto issue_w = [&](int p, int q, int slot) {
      const int sow = (p * KS + q * 128) * N;
      LAS char* lwp = (LAS char*)smem + slot * SLOT + XB + lw * 32 * 128;
#pragma unroll
      for (int i = 0; i < 4; ++i)
        __builtin_amdgcn_raw_ptr_buffer_load_lds(rw, lwp + i * 1024, 16, (i & 2) ? vw1 : vw0, sow + i * 8 * N, 0, 0);
    };
    auto issue_local = [&](int s, int slot) {
      issue_w(rank, s, slot);
      LAS char* lx = (LAS char*)smem + slot * SLOT + lw * 64 * 128;
#pragma unroll
      for (int i = 0; i < 8; ++i)
        __builtin_amdgcn_raw_ptr_buffer_load_lds(rxl, lx + i * 1024, 16, (i & 1) ? vx1 : vx0, s * 128 + i * 8 * KS, 0, 0);
    };
    issue_local(0, 0); issue_local(1, 1);
    if (lw == 0 && lane < 40 && lane != 1) lctl[lane] = 0u;
    asm volatile("s_waitcnt vmcnt(12) lgkmcnt(0)" ::: "memory");
    BAR();
    issue_local(2, 2);
    LTRACE(32);
    STAMP(1);
    const unsigned call = __builtin_amdgcn_readfirstlane(lctl[1]);
    __amdgpu_buffer_rsrc_t rxs = __builtin_amdgcn_make_buffer_rsrc((void*)(myws + WS_X + (size_t)(call & 1u) * NR * BLK), 0, NR * BLK, 0x00020000);
    __amdgpu_buffer_rsrc_t rfl = __builtin_amdgcn_make_buffer_rsrc((void*)(myws + WS_FLAG + mh * 2048), 0, 2048, 0x00020000);
    unsigned flg_addr, my_entry;
    {
      int p, q; piece_pq(min(lane >> 1, NREM - 1), rank, p, q);
      flg_addr = (unsigned)(size_t)((LAS char*)smem + LDS_FLG) + (p * 4 + q) * 64 + (lane & 1) * 32;
      my_entry = 0x100u | (unsigned)(p * 4 + q);
    }
    auto poll_issue = [&]() {
      __builtin_amdgcn_raw_ptr_buffer_load_lds(rfl, (LAS char*)smem + LDS_FLG, 16, lane * 16, 0, 0, 17);
      __builtin_amdgcn_raw_ptr_buffer_load_lds(rfl, (LAS char*)smem + LDS_FLG + 1024, 16, lane * 16, 1024, 0, 17);
    };
    auto poll_eval = [&]() -> unsigned long long {
      const v4u f0 = *(volatile LAS v4u*)(size_t)flg_addr, f1 = *(volatile LAS v4u*)(size_t)(flg_addr + 16);
      const unsigned m = min(min(min(f0.x, f0.y), min(f0.z, f0.w)), min(min(f1.x, f1.y), min(f1.z, f1.w)));
      const unsigned long long ok = __ballot(m >= call && lane < 2 * NREM);
      return ok & (ok >> 1) & 0x5555555555555555ull;
    };
    unsigned long long ready = 0, used = 0;
    constexpr unsigned long long ALL = 0x0055555555555555ull;
    int nsched = 4;
    auto schedule = [&](int upto) {
      const unsigned long long avail = ready & ~used;
      const int q0 = (__builtin_ctzll(ALL & ~used) >> 1) / 7;
      const unsigned long long gq = 0x1555ull << (14 * q0);
      const unsigned long long first = ready == ALL ? avail : (avail & gq), rest = avail & ~first;
      const int nfirst = __builtin_popcountll(first);
      const bool in_first = (first >> lane) & 1ull;
      const unsigned long long m = in_first ? first : rest;
      const int pos = nsched + (in_first ? 0 : nfirst) + (int)__builtin_amdgcn_mbcnt_hi((unsigned)(m >> 32), __builtin_amdgcn_mbcnt_lo((unsigned)m, 0u));
      const bool take = ((avail >> lane) & 1ull) && (in_first || pos <= upto);
      if (take) lctl[4 + pos] = my_entry;
      const unsigned long long taken = __ballot(take);
      used |= taken;
      nsched += __builtin_popcountll(taken);
    };
    auto issue_remote = [&](unsigned e, int slot) {
      const int pq = (int)(e & 31u);
      LAS char* lwp = (LAS char*)smem + slot * SLOT + XB + lw * 32 * 128;
      const int sow = pq * 128 * N;
#pragma unroll
      for (int i = 0; i < 4; ++i)
        __builtin_amdgcn_raw_ptr_buffer_load_lds(rw, lwp + i * 1024, 16, (i & 2) ? vw1 : vw0, sow + i * 8 * N, 0, 0);
      LAS char* lx = (LAS char*)smem + slot * SLOT + lw * 64 * 128;
      const int sox = pq * (MS * 128);
#pragma unroll
      for (int i = 0; i < 8; ++i)
        __builtin_amdgcn_raw_ptr_buffer_load_lds(rxs, lx + i * 1024, 16, (i & 1) ? vs1 : vs0, sox + i * 1024, 0, 17);
    };
    asm volatile("s_waitcnt vmcnt(12)" ::: "memory");
    BAR();
    LTRACE(0);
    int poll_age = 0;
    if (FIRST_LOOK && lw == 0 && ready != ALL) { poll_issue(); poll_age = 3; }
    issue_local(3, 0);
    int slot = 1;
    int t = 1;
    unsigned tbl = 0;
    for (; t < NSTEP - 3; ++t) {
      if (poll_age == 1 || poll_age == 2) asm volatile("s_waitcnt vmcnt(14)" ::: "memory");
      else asm volatile("s_waitcnt vmcnt(12)" ::: "memory");
      const int s = t + 3;
      if (lw == 0) {
        if (poll_age == 2) {
          unsigned sts; asm volatile("s_getreg_b32 %0, hwreg(HW_REG_IB_STS)" : "=s"(sts));
          if (((sts & 15u) | ((sts >> 18) & 0x30u)) <= 12u) { asm volatile("s_waitcnt vmcnt(12)" ::: "memory"); poll_age = 3; }
        }
        bool news = false;
        if (poll_age == 3) { ready |= poll_eval(); poll_age = 0; news = true; }
        if ((news || nsched <= s + 2) && (ready & ~used)) schedule(s + 2);
      }
      unsigned e = __builtin_amdgcn_readlane(tbl, s);
      BAR();
      LTRACE(t);
      if (!e) {
        tbl = lctl[4 + (lane & 31)];
        e = __builtin_amdgcn_readlane(tbl, s);
        if (!e) {
          if (lw == 0) {
            const unsigned long long t0 = __builtin_amdgcn_s_memrealtime();
            for (unsigned n = 1;; ++n) {
              poll_issue();
              asm volatile("s_waitcnt vmcnt(0)" ::: "memory");
              ready |= poll_eval();
              if (ready & ~used) break;
              if ((n & 15u) == 0u) {
                const unsigned tv = __builtin_amdgcn_readfirstlane(__hip_atomic_load(tmo, __ATOMIC_RELAXED, __HIP_MEMORY_SCOPE_AGENT));
                if (tv == call || __builtin_amdgcn_s_memrealtime() - t0 > DEADLINE_TICKS) {
                  if (lane == 0) { __hip_atomic_store(tmo, call, __ATOMIC_RELAXED, __HIP_MEMORY_SCOPE_AGENT); lctl[0] = 1u; }
                  ready = ALL;
                  break;
                }
              }
            }
            poll_age = 0;
            schedule(s + 2);
          }
          for (;;) {
            tbl = lctl[4 + (lane & 31)];
            e = __builtin_amdgcn_readlane(tbl, s);
            if (e) break;
            __builtin_amdgcn_s_sleep(1);
          }
        }
      }
      const unsigned tbl_next = lctl[4 + (lane & 31)];
      const int have = (lw == 0 && ready != ALL && poll_age == 0) ? nsched - s - 1 + __builtin_popcountll(ready & ~used) : 99;
      if (have < POLL_URGENT) poll_issue();
      issue_remote(e, slot);
      if (poll_age) ++poll_age;
      else if (have < POLL_URGENT) poll_age = 3;
      else if (have < POLL_LOW || (have < 99 && (t & POLL_EVERY) == 0)) { poll_issue(); poll_age = 1; }
      tbl = tbl_next;
      slot = (slot == 2) ? 0 : slot + 1;
      if (__builtin_amdgcn_readlane(tbl, NSTEP - 1) != 0u && poll_age == 0) { ++t; break; }
    }
    for (; t < NSTEP - 3; ++t) {
      asm volatile("s_waitcnt vmcnt(12)" ::: "memory");
      const unsigned e = __builtin_amdgcn_readlane(tbl, t + 3);
      BAR();
      LTRACE(t);
      issue_remote(e, slot);
      slot = (slot == 2) ? 0 : slot + 1;
    }
    asm volatile("s_waitcnt vmcnt(12)" ::: "memory");
    BAR();
    LTRACE(29);
    asm volatile("s_waitcnt vmcnt(0)" ::: "memory");
    BAR();
    LTRACE(30);
    BAR();
    LTRACE(31);
    STAMP_FLUSH(12);
    return;
  }

  const int dst = (grp + idx) & 7;
  const bool pushes = (dst != rank);
  v4i px0, px1;
  const int poff = (idx * 16 + (lane >> 3)) * KS + wid * 128 + (lane & 7) * 16;
  char* const dws = P.ws[dst];
  unsigned pf_dummy = 0;
  unsigned call;
  {
    const char* src = P.x + (size_t)dst * BLK + poff;
    if (wid == 0) {
      unsigned* cp = (unsigned*)(myws + WS_CTR) + b * 32;
      unsigned tick = 0, one = 1u;
      asm volatile("s_mov_b64 exec, 1\n\tglobal_atomic_add %0, %3, %4, off sc0\n\ts_mov_b64 exec, -1\n\t"
                   "global_load_dwordx4 %1, %5, off\n\tglobal_load_dwordx4 %2, %6, off\n\ts_waitcnt vmcnt(0)"
                   : "+v"(tick), "=&v"(px0), "=&v"(px1) : "v"(cp), "v"(one), "v"(src), "v"(src + 8 * KS) : "memory");
      call = __builtin_amdgcn_readfirstlane(tick) + 1u;
      if (lane == 0) lctl[1] = call;
    } else {
      asm volatile("global_load_dwordx4 %0, %2, off\n\tglobal_load_dwordx4 %1, %3, off\n\ts_waitcnt vmcnt(0)"
                   : "=&v"(px0), "=&v"(px1) : "v"(src), "v"(src + 8 * KS) : "memory");
      call = 0;
    }
  }
  STAMP(1);
  char* const pdst0 = dws + WS_X + (size_t)rank * BLK + wid * (MS * 128) + (idx * 16 + (lane >> 3)) * 128 + (lane & 7) * 16;
  unsigned* const pflag = (unsigned*)(dws + WS_FLAG) + (((idx >> 4) * 8 + rank) * 4 + wid) * 16 + (idx & 15);
  auto push_stores = [&]() {
    char* const pdst = pdst0 + (size_t)(call & 1u) * NR * BLK;
    asm volatile("global_store_dwordx4 %0, %1, off sc0 sc1\n\ts_nop 1" :: "v"(pdst), "v"(px0) : "memory");
    asm volatile("global_store_dwordx4 %0, %1, off offset:1024 sc0 sc1\n\ts_nop 1" :: "v"(pdst), "v"(px1) : "memory");
  };
  const int pj = (rank - 1 - dst) & 7;
  const int t_store = T_STORE(wid, pj);
  const int t_force = (t_store < 0 ? -1 : t_store) + 3;
  int pstate = pushes ? 0 : 2;
  if (pushes && t_store < 0) { push_stores(); pstate = 1; }
  STAMP(10);
  if (pstate == 1) { asm volatile("s_waitcnt vmcnt(0)" ::: "memory"); STAMP(11); if (lane == 0) __hip_atomic_store(pflag, call, __ATOMIC_RELAXED, __HIP_MEMORY_SCOPE_SYSTEM); pstate = 2; }
  int t_evt = pstate == 2 ? NSTEP : (pstate == 1 ? 0 : t_store);
  int t_mid = pstate == 1 ? 0 : NSTEP;
  auto push_event = [&](int t, bool force) {
    if (pstate == 0) { push_stores(); pstate = 1; if (wid == 0) t_mid = t; }
    else {
      unsigned sts; asm volatile("s_getreg_b32 %0, hwreg(HW_REG_IB_STS)" : "=s"(sts));
      if (force || (sts & 0x00C0000Fu) == 0u) {
        asm volatile("s_waitcnt vmcnt(0)" ::: "memory");
        if (lane == 0) __hip_atomic_store(pflag, call, __ATOMIC_RELAXED, __HIP_MEMORY_SCOPE_SYSTEM);
        pstate = 2; t_evt = NSTEP; t_mid = NSTEP;
      }
    }
  };

  const int wm = wid >> 1, wn = wid & 1;
  const int g = lane >> 4, l15 = lane & 15;
  const int fx = (l15 >> 1) & 7;
  const unsigned ax0 = (wm * 128 + l15) * 128 + ((g ^ fx) << 4);
  const unsigned ax1 = (wm * 128 + l15) * 128 + (((4 + g) ^ fx) << 4);
  const int q8 = l15 >> 1;
  unsigned aw[4];
#pragma unroll
  for (int j = 0; j < 4; ++j) aw[j] = XB + (16 * g + q8) * 128 + (((4 * wn + j) ^ (((q8 >> 1) & 3) | ((g & 1) << 2))) << 4) + 8 * (lane & 1);
  v4f acc[8][4];
#pragma unroll
  for (int i = 0; i < 8; ++i)
#pragma unroll
    for (int j = 0; j < 4; ++j) acc[i][j] = (v4f){};
  v8i alo[4], ahi[4], bf[4];
  const unsigned sb0 = (unsigned)(size_t)(LAS char*)smem;
#define RDA(dst, i, ii, sb) dst[i] = cat4(*(LAS v4i*)(size_t)(ax0 + (sb) + (ii) * 2048), *(LAS v4i*)(size_t)(ax1 + (sb) + (ii) * 2048))
#define TR(j, t, sb) __builtin_amdgcn_ds_read_tr8_b64_v2i32((LAS v2i*)(size_t)(aw[j] + (sb) + ((t) >> 1) * 8192 + ((t) & 1) * 1024))
#define RDB(j, sb) bf[j] = cat2(TR(j, 0, sb), TR(j, 1, sb), TR(j, 2, sb), TR(j, 3, sb))
#define MM(i, j, A, ai) acc[i][j] = __builtin_amdgcn_mfma_scale_f32_16x16x128_f8f6f4(A[ai], bf[j], acc[i][j], 1, 1, 0, 0, 0, 0)
#define MM4(i0, j, A) do { MM(i0, j, A, 0); MM(i0 + 1, j, A, 1); MM(i0 + 2, j, A, 2); MM(i0 + 3, j, A, 3); } while (0)
#define SGB(m, n) __builtin_amdgcn_sched_group_barrier(m, n, 0)
#define P1(n) SGB(0x8,1); SGB(0x100,n);
  WAIT_LGKM0();
  BAR();
  STAMP(2);
  call = __builtin_amdgcn_readfirstlane(lctl[1]);
  {
    const unsigned sb = sb0;
    RDA(alo, 0, 0, sb); RDA(alo, 1, 1, sb); RDA(alo, 2, 2, sb); RDA(alo, 3, 3, sb); RDB(0, sb); RDB(1, sb); RDB(2, sb);
  }
  unsigned sb = sb0;
  int slot = 0;
  const unsigned pf_voff = (4 * idx + (lane >> 4)) * N + grp * 2048 + (lane & 15) * 128;
  int pf_turn = 1, pf_q = (PF_D - 1) / 7, pf_i = (PF_D - 1) % 7;
  unsigned pf_em = PF_START == 0 ? (unsigned)(((1 ^ wid) - 1) >> 31) : 0u;
  const char* pf_ptr = P.w + (size_t)(unsigned)(((rank + 1 + pf_i) & 7) * KS + pf_q * 128) * N;
#pragma nounroll
  for (int t = 0; t < NSTEP; ++t) {
    const int nslot = (slot == 2) ? 0 : slot + 1;
    const unsigned sbn = sb0 + nslot * SLOT;
    SCHED();
    RDB(3, sb);
    MM(0, 0, alo, 0);
    P1(4)
    SCHED();
    if (t >= t_evt) push_event(t, t >= t_force);
    asm volatile("s_mov_b32 exec_lo, %3\n\ts_mov_b32 exec_hi, %3\n\tglobal_load_dword %0, %1, %2\n\ts_mov_b64 exec, -1" : "+v"(pf_dummy) : "v"(pf_voff), "s"(pf_ptr), "s"(__builtin_amdgcn_readfirstlane(pf_em)) : "memory");
    SCHED();
    RDA(ahi, 0, 4, sb); RDA(ahi, 1, 5, sb); RDA(ahi, 2, 6, sb); RDA(ahi, 3, 7, sb);
    MM(1, 0, alo, 1); MM(2, 0, alo, 2); MM(3, 0, alo, 3);
    MM4(0, 1, alo); MM4(0, 2, alo); MM4(0, 3, alo);
    P1(1) P1(1) P1(1) P1(1) P1(1) P1(1) P1(1) P1(1) SGB(0x8, 7);
    SCHED();
    WAIT_LGKM0();
    BAR();
    SCHED();
    RDA(alo, 0, 0, sbn);
    MM(4, 0, ahi, 0);
    P1(2)
    SCHED();
    if (t >= t_mid) push_event(t, false);
    pf_turn = pf_turn == 3 ? 1 : pf_turn + 1;
    if (++pf_i == 7) { pf_i = 0; ++pf_q; }
    pf_em = (unsigned)(((t + 4 + PF_D - NSTEP) >> 31) & ((PF_START - t - 2) >> 31) & (((pf_turn ^ wid) - 1) >> 31));
    pf_ptr = P.w + (size_t)(unsigned)(((rank + 1 + pf_i) & 7) * KS + pf_q * 128) * N;
    SCHED();
    RDA(alo, 1, 1, sbn); RDA(alo, 2, 2, sbn); RDA(alo, 3, 3, sbn);
    MM(5, 0, ahi, 1); MM(6, 0, ahi, 2); MM(7, 0, ahi, 3);
    P1(2) P1(2) P1(2)
    SCHED();
    RDB(0, sbn);
    MM4(4, 1, ahi);
    P1(1) P1(1) P1(1) P1(1)
    SCHED();
    RDB(1, sbn);
    MM4(4, 2, ahi);
    P1(1) P1(1) P1(1) P1(1)
    SCHED();
    RDB(2, sbn);
    MM4(4, 3, ahi);
    P1(1) P1(1) P1(1) P1(1)
    SCHED();
    sb = sbn; slot = nslot;
  }
  STAMP(6);
  asm volatile("s_waitcnt vmcnt(0)" : "+v"(pf_dummy) :: "memory");
  const unsigned tmo_seen = __hip_atomic_load(tmo, __ATOMIC_RELAXED, __HIP_MEMORY_SCOPE_AGENT);
  const bool bad = (tmo_seen == call) || (lctl[0] != 0u);
  const float s = bad ? __builtin_nanf("") : P.sx[0] * P.sw[0];
  __amdgpu_buffer_rsrc_t ro = __builtin_amdgcn_make_buffer_rsrc((void*)P.out, 0, MS * N * 4, 0x00020000);
  const int vo = ((m0 + wm * 128 + 8 * (lane >> 5)) * N + n0 + wn * 64 + (lane & 31)) * 4;
#pragma unroll
  for (int i = 0; i < 8; ++i)
#pragma unroll
    for (int r = 0; r < 4; ++r)
#pragma unroll
      for (int jp = 0; jp < 2; ++jp) {
        unsigned o[2];
#pragma unroll
        for (int jj = 0; jj < 2; ++jj) {
          const float y = acc[i][2 * jp + jj][r] * s;
          const float c = __builtin_amdgcn_fmed3f(y, -60.f, 60.f);
          const float e = __builtin_amdgcn_exp2f(c * -1.44269504088896f);
          o[jj] = __builtin_bit_cast(unsigned, y * __builtin_amdgcn_rcpf(1.0f + e));
        }
        const auto sw2 = __builtin_amdgcn_permlane16_swap(o[0], o[1], false, false);
        __builtin_amdgcn_raw_buffer_store_b32(sw2[0], ro, vo + jp * 128, (16 * i + r) * N * 4, 16);
        __builtin_amdgcn_raw_buffer_store_b32(sw2[1], ro, vo + jp * 128, (16 * i + r + 4) * N * 4, 16);
      }
}

extern "C" void kernel_launch(int rank, int nranks, ncclComm_t comm,
                              hipStream_t stream,
                              void* const* d_in, const int* in_sizes, int n_in,
                              void* d_out, int out_size,
                              const int* in_dtypes, int out_dtype,
                              void* const* d_out_all,
                              void* const* d_ws_all, size_t ws_bytes) {
  (void)comm; (void)in_dtypes; (void)out_dtype; (void)d_out_all;
  if (nranks != NR || rank < 0 || rank >= NR || n_in != 4 || in_sizes[0] != NR * BLK || in_sizes[1] != K * N ||
      out_size != MS * N || ws_bytes < WS_NEED) {
    fprintf(stderr, "kernel_launch: unexpected problem shape\n");
    return;
  }
  static bool attr_set = false;
  if (!attr_set) {
    HIP_CHECK(hipFuncSetAttribute((const void*)a2a_gemm_k, hipFuncAttributeMaxDynamicSharedMemorySize, LDS_BYTES));
    attr_set = true;
  }
  Params P;
  P.x = (const char*)d_in[0]; P.w = (const char*)d_in[1]; P.sx = (const float*)d_in[2]; P.sw = (const float*)d_in[3];
  P.out = (float*)d_out; P.myws = (char*)d_ws_all[rank];
  for (int p = 0; p < NR; ++p) P.ws[p] = (char*)d_ws_all[p];
  P.rank = rank; P.pad = 0;
  a2a_gemm_k<<<256, 512, LDS_BYTES, stream>>>(P);
  HIP_CHECK(hipGetLastError());
}
```

```cpp
#include <hip/hip_runtime.h>
#include <rccl/rccl.h>
#include <cstdio>

#define HIP_CHECK(expr) do { hipError_t _e = (expr); if (_e != hipSuccess) { \
    fprintf(stderr, "HIP error %s at %s:%d\n", hipGetErrorString(_e), __FILE__, __LINE__); \
    return; } } while (0)

#define LAS __attribute__((address_space(3)))
typedef int v2i __attribute__((ext_vector_type(2)));
typedef int v4i __attribute__((ext_vector_type(4)));
typedef int v8i __attribute__((ext_vector_type(8)));
typedef unsigned v4u __attribute__((ext_vector_type(4)));
typedef float v4f __attribute__((ext_vector_type(4)));

constexpr int NR = 8;
constexpr int MS = 512, N = 16384, K = 4096;
constexpr int KS = K / NR;
constexpr int BLK = MS * KS;
constexpr int SLOT = 49152, XB = 32768;
constexpr int NSLOT = 3, NSTEP = 32, NREM = 28;
constexpr int LDS_CTRL = NSLOT * SLOT;
constexpr int LDS_FLG = LDS_CTRL + 256;
constexpr int LDS_SCRAP = LDS_FLG + 2048 + 256;
constexpr int LDS_BYTES = LDS_SCRAP + 256;
constexpr size_t WS_CTR = 0;
constexpr size_t WS_TMO = 32768;
constexpr size_t WS_FLAG = 36864;
constexpr size_t WS_X = 1 << 20;
constexpr size_t WS_NEED = WS_X + 2 * (size_t)NR * BLK;
constexpr unsigned long long DEADLINE_TICKS = 100000000ull;

#define POLL_LOW 5
#define POLL_EVERY 3
#define POLL_URGENT 3
#define FIRST_LOOK 0
#define T_STORE(w, j) ((w) == 0 ? -1 : 2 * ((w) - 1))
#define PFA_D 4
#define PFA_N0 1

struct Params {
  const char* x; const char* w; const float* sx; const float* sw; float* out; char* myws;
  char* ws[NR];
  int rank; int pad;
};

#define LTRACE(i) do {} while (0)
#define STAMP(i) do {} while (0)
#define STAMP_FLUSH(n) do {} while (0)

__device__ __forceinline__ v8i cat4(v4i lo, v4i hi) { return __builtin_shufflevector(lo, hi, 0, 1, 2, 3, 4, 5, 6, 7); }
__device__ __forceinline__ v8i cat2(v2i a, v2i b, v2i c, v2i d) {
  v4i lo = __builtin_shufflevector(a, b, 0, 1, 2, 3), hi = __builtin_shufflevector(c, d, 0, 1, 2, 3);
  return cat4(lo, hi);
}
#define WAIT_LGKM0() __builtin_amdgcn_s_waitcnt(0xC07F)
#define BAR() __builtin_amdgcn_s_barrier()
#define SCHED() __builtin_amdgcn_sched_barrier(0)

__device__ __forceinline__ void piece_pq(int u, int rank, int& p, int& q) { q = u / 7; p = (rank + 1 + (u - q * 7)) & 7; }

__global__ __launch_bounds__(512, 2) void a2a_gemm_k(const Params P) {
  extern __shared__ __attribute__((aligned(16))) char smem[];
  const int tid = threadIdx.x, wid = __builtin_amdgcn_readfirstlane(tid >> 6), lane = tid & 63;
  const int b = blockIdx.x, grp = b & 7, idx = b >> 3;
  const int mh = idx & 1, nt = grp * 16 + (idx >> 1);
  const int m0 = mh * 256, n0 = nt * 128;
  const int rank = P.rank;
  char* const myws = P.myws;
  STAMP(0);
  unsigned* const tmo = (unsigned*)(myws + WS_TMO);
  volatile LAS unsigned* const lctl = (volatile LAS unsigned*)((LAS char*)smem + LDS_CTRL);

  if (wid >= 4) {
    const int lw = wid - 4;
    __amdgpu_buffer_rsrc_t rxl = __builtin_amdgcn_make_buffer_rsrc((void*)(P.x + (size_t)rank * BLK), 0, BLK, 0x00020000);
    __amdgpu_buffer_rsrc_t rw = __builtin_amdgcn_make_buffer_rsrc((void*)P.w, 0, K * N, 0x00020000);
    const int r8 = lane >> 3, sl = lane & 7;
    const int vx0 = (m0 + lw * 64 + r8) * KS + ((sl ^ (r8 >> 1)) << 4);
    const int vx1 = (m0 + lw * 64 + r8) * KS + ((sl ^ (4 + (r8 >> 1))) << 4);
    const int vs0 = (m0 + lw * 64 + r8) * 128 + ((sl ^ (r8 >> 1)) << 4);
    const int vs1 = (m0 + lw * 64 + r8) * 128 + ((sl ^ (4 + (r8 >> 1))) << 4);
    const int vw0 = (lw * 32 + r8) * N + n0 + ((sl ^ ((r8 >> 1) & 3)) << 4);
    const int vw1 = (lw * 32 + r8) * N + n0 + ((sl ^ (((r8 >> 1) & 3) | 4)) << 4);
    auto issue_w = [&](int p, int q, int slot) {
      const int sow = (p * KS + q * 128) * N;
      LAS char* lwp = (LAS char*)smem + slot * SLOT + XB + lw * 32 * 128;
#pragma unroll
      for (int i = 0; i < 4; ++i)
        __builtin_amdgcn_raw_ptr_buffer_load_lds(rw, lwp + i * 1024, 16, (i & 2) ? vw1 : vw0, sow + i * 8 * N, 0, 0);
    };
    auto issue_local = [&](int s, int slot) {
      issue_w(rank, s, slot);
      LAS char* lx = (LAS char*)smem + slot * SLOT + lw * 64 * 128;
#pragma unroll
      for (int i = 0; i < 8; ++i)
        __builtin_amdgcn_raw_ptr_buffer_load_lds(rxl, lx + i * 1024, 16, (i & 1) ? vx1 : vx0, s * 128 + i * 8 * KS, 0, 0);
    };
    const bool pfw = PFA_D > 0 && lw == 3;
    const int pf_voff = (4 * idx + (lane >> 4)) * N + grp * 2048 + (lane & 15) * 128;
    auto prefetch = [&](int u) {
      int p, q; piece_pq(u < NREM ? u : NREM - 1, rank, p, q);
      __builtin_amdgcn_raw_ptr_buffer_load_lds(rw, (LAS char*)smem + LDS_SCRAP, 4, pf_voff, (p * KS + q * 128) * N, 0, 0);
    };
    issue_local(0, 0); issue_local(1, 1);
    if (pfw) {
#pragma unroll
      for (int u = 0; u < PFA_N0; ++u) prefetch(u);
    }
    if (lw == 0 && lane < 40 && lane != 1) lctl[lane] = 0u;
    if (pfw) asm volatile("s_waitcnt vmcnt(%0) lgkmcnt(0)" :: "n"(12 + PFA_N0) : "memory");
    else asm volatile("s_waitcnt vmcnt(12) lgkmcnt(0)" ::: "memory");
    BAR();
    issue_local(2, 2);
    LTRACE(32);
    STAMP(1);
    const unsigned call = __builtin_amdgcn_readfirstlane(lctl[1]);
    __amdgpu_buffer_rsrc_t rxs = __builtin_amdgcn_make_buffer_rsrc((void*)(myws + WS_X + (size_t)(call & 1u) * NR * BLK), 0, NR * BLK, 0x00020000);
    __amdgpu_buffer_rsrc_t rfl = __builtin_amdgcn_make_buffer_rsrc((void*)(myws + WS_FLAG + mh * 2048), 0, 2048, 0x00020000);
    unsigned flg_addr, my_entry;
    {
      int p, q; piece_pq(min(lane >> 1, NREM - 1), rank, p, q);
      flg_addr = (unsigned)(size_t)((LAS char*)smem + LDS_FLG) + (p * 4 + q) * 64 + (lane & 1) * 32;
      my_entry = 0x100u | (unsigned)(p * 4 + q);
    }
    auto poll_issue = [&]() {
      __builtin_amdgcn_raw_ptr_buffer_load_lds(rfl, (LAS char*)smem + LDS_FLG, 16, lane * 16, 0, 0, 17);
      __builtin_amdgcn_raw_ptr_buffer_load_lds(rfl, (LAS char*)smem + LDS_FLG + 1024, 16, lane * 16, 1024, 0, 17);
    };
    auto poll_eval = [&]() -> unsigned long long {
      const v4u f0 = *(volatile LAS v4u*)(size_t)flg_addr, f1 = *(volatile LAS v4u*)(size_t)(flg_addr + 16);
      const unsigned m = min(min(min(f0.x, f0.y), min(f0.z, f0.w)), min(min(f1.x, f1.y), min(f1.z, f1.w)));
      const unsigned long long ok = __ballot(m >= call && lane < 2 * NREM);
      return ok & (ok >> 1) & 0x5555555555555555ull;
    };
    unsigned long long ready = 0, used = 0;
    constexpr unsigned long long ALL = 0x0055555555555555ull;
    int nsched = 4;
    auto schedule = [&](int upto) {
      const unsigned long long avail = ready & ~used;
      const int q0 = (__builtin_ctzll(ALL & ~used) >> 1) / 7;
      const unsigned long long gq = 0x1555ull << (14 * q0);
      const unsigned long long first = ready == ALL ? avail : (avail & gq), rest = avail & ~first;
      const int nfirst = __builtin_popcountll(first);
      const bool in_first = (first >> lane) & 1ull;
      const unsigned long long m = in_first ? first : rest;
      const int pos = nsched + (in_first ? 0 : nfirst) + (int)__builtin_amdgcn_mbcnt_hi((unsigned)(m >> 32), __builtin_amdgcn_mbcnt_lo((unsigned)m, 0u));
      const bool take = ((avail >> lane) & 1ull) && (in_first || pos <= upto);
      if (take) lctl[4 + pos] = my_entry;
      const unsigned long long taken = __ballot(take);
      used |= taken;
      nsched += __builtin_popcountll(taken);
    };
    auto issue_remote = [&](unsigned e, int slot) {
      const int pq = (int)(e & 31u);
      LAS char* lwp = (LAS char*)smem + slot * SLOT + XB + lw * 32 * 128;
      const int sow = pq * 128 * N;
#pragma unroll
      for (int i = 0; i < 4; ++i)
        __builtin_amdgcn_raw_ptr_buffer_load_lds(rw, lwp + i * 1024, 16, (i & 2) ? vw1 : vw0, sow + i * 8 * N, 0, 0);
      LAS char* lx = (LAS char*)smem + slot * SLOT + lw * 64 * 128;
      const int sox = pq * (MS * 128);
#pragma unroll
      for (int i = 0; i < 8; ++i)
        __builtin_amdgcn_raw_ptr_buffer_load_lds(rxs, lx + i * 1024, 16, (i & 1) ? vs1 : vs0, sox + i * 1024, 0, 17);
    };
    if (pfw) asm volatile("s_waitcnt vmcnt(%0)" :: "n"(12 + PFA_N0) : "memory");
    else asm volatile("s_waitcnt vmcnt(12)" ::: "memory");
    BAR();
    LTRACE(0);
    int poll_age = 0;
    if (FIRST_LOOK && lw == 0 && ready != ALL) { poll_issue(); poll_age = 3; }
    issue_local(3, 0);
    int slot = 1;
    int t = 1;
    unsigned tbl = 0;
    for (; t < NSTEP - 3; ++t) {
      if (pfw && t >= 3) asm volatile("s_waitcnt vmcnt(%0)" :: "n"(14) : "memory");
      else if (pfw && t == 2) asm volatile("s_waitcnt vmcnt(%0)" :: "n"(13) : "memory");
      else if (poll_age == 1 || poll_age == 2) asm volatile("s_waitcnt vmcnt(14)" ::: "memory");
      else asm volatile("s_waitcnt vmcnt(12)" ::: "memory");
      const int s = t + 3;
      if (lw == 0) {
        if (poll_age == 2) {
          unsigned sts; asm volatile("s_getreg_b32 %0, hwreg(HW_REG_IB_STS)" : "=s"(sts));
          if (((sts & 15u) | ((sts >> 18) & 0x30u)) <= 12u) { asm volatile("s_waitcnt vmcnt(12)" ::: "memory"); poll_age = 3; }
        }
        bool news = false;
        if (poll_age == 3) { ready |= poll_eval(); poll_age = 0; news = true; }
        if ((news || nsched <= s + 2) && (ready & ~used)) schedule(s + 2);
      }
      unsigned e = __builtin_amdgcn_readlane(tbl, s);
      BAR();
      LTRACE(t);
      if (!e) {
        tbl = lctl[4 + (lane & 31)];
        e = __builtin_amdgcn_readlane(tbl, s);
        if (!e) {
          if (lw == 0) {
            const unsigned long long t0 = __builtin_amdgcn_s_memrealtime();
            for (unsigned n = 1;; ++n) {
              poll_issue();
              asm volatile("s_waitcnt vmcnt(0)" ::: "memory");
              ready |= poll_eval();
              if (ready & ~used) break;
              if ((n & 15u) == 0u) {
                const unsigned tv = __builtin_amdgcn_readfirstlane(__hip_atomic_load(tmo, __ATOMIC_RELAXED, __HIP_MEMORY_SCOPE_AGENT));
                if (tv == call || __builtin_amdgcn_s_memrealtime() - t0 > DEADLINE_TICKS) {
                  if (lane == 0) { __hip_atomic_store(tmo, call, __ATOMIC_RELAXED, __HIP_MEMORY_SCOPE_AGENT); lctl[0] = 1u; }
                  ready = ALL;
                  break;
                }
              }
            }
            poll_age = 0;
            schedule(s + 2);
          }
          for (;;) {
            tbl = lctl[4 + (lane & 31)];
            e = __builtin_amdgcn_readlane(tbl, s);
            if (e) break;
            __builtin_amdgcn_s_sleep(1);
          }
        }
      }
      const unsigned tbl_next = lctl[4 + (lane & 31)];
      const int have = (lw == 0 && ready != ALL && poll_age == 0) ? nsched - s - 1 + __builtin_popcountll(ready & ~used) : 99;
      if (have < POLL_URGENT) poll_issue();
      issue_remote(e, slot);
      if (pfw) prefetch(t - 1 + PFA_D);
      if (poll_age) ++poll_age;
      else if (have < POLL_URGENT) poll_age = 3;
      else if (have < POLL_LOW || (have < 99 && (t & POLL_EVERY) == 0)) { poll_issue(); poll_age = 1; }
      tbl = tbl_next;
      slot = (slot == 2) ? 0 : slot + 1;
      if (__builtin_amdgcn_readlane(tbl, NSTEP - 1) != 0u && poll_age == 0) { ++t; break; }
    }
    for (; t < NSTEP - 3; ++t) {
      if (pfw && t >= 3) asm volatile("s_waitcnt vmcnt(%0)" :: "n"(14) : "memory");
      else if (pfw && t == 2) asm volatile("s_waitcnt vmcnt(%0)" :: "n"(13) : "memory");
      else asm volatile("s_waitcnt vmcnt(12)" ::: "memory");
      const unsigned e = __builtin_amdgcn_readlane(tbl, t + 3);
      BAR();
      LTRACE(t);
      issue_remote(e, slot);
      if (pfw) prefetch(t - 1 + PFA_D);
      slot = (slot == 2) ? 0 : slot + 1;
    }
    if (pfw) asm volatile("s_waitcnt vmcnt(%0)" :: "n"(14) : "memory");
    else asm volatile("s_waitcnt vmcnt(12)" ::: "memory");
    BAR();
    LTRACE(29);
    asm volatile("s_waitcnt vmcnt(0)" ::: "memory");
    BAR();
    LTRACE(30);
    BAR();
    LTRACE(31);
    STAMP_FLUSH(12);
    return;
  }

  const int dst = (grp + idx) & 7;
  const bool pushes = (dst != rank);
  v4i px0, px1;
  const int poff = (idx * 16 + (lane >> 3)) * KS + wid * 128 + (lane & 7) * 16;
  char* const dws = P.ws[dst];
  unsigned call;
  {
    const char* src = P.x + (size_t)dst * BLK + poff;
    if (wid == 0) {
      unsigned* cp = (unsigned*)(myws + WS_CTR) + b * 32;
      unsigned tick = 0, one = 1u;
      asm volatile("s_mov_b64 exec, 1\n\tglobal_atomic_add %0, %3, %4, off sc0\n\ts_mov_b64 exec, -1\n\t"
                   "global_load_dwordx4 %1, %5, off\n\tglobal_load_dwordx4 %2, %6, off\n\ts_waitcnt vmcnt(0)"
                   : "+v"(tick), "=&v"(px0), "=&v"(px1) : "v"(cp), "v"(one), "v"(src), "v"(src + 8 * KS) : "memory");
      call = __builtin_amdgcn_readfirstlane(tick) + 1u;
      if (lane == 0) lctl[1] = call;
    } else {
      asm volatile("global_load_dwordx4 %0, %2, off\n\tglobal_load_dwordx4 %1, %3, off\n\ts_waitcnt vmcnt(0)"
                   : "=&v"(px0), "=&v"(px1) : "v"(src), "v"(src + 8 * KS) : "memory");
      call = 0;
    }
  }
  STAMP(1);
  char* const pdst0 = dws + WS_X + (size_t)rank * BLK + wid * (MS * 128) + (idx * 16 + (lane >> 3)) * 128 + (lane & 7) * 16;
  unsigned* const pflag = (unsigned*)(dws + WS_FLAG) + (((idx >> 4) * 8 + rank) * 4 + wid) * 16 + (idx & 15);
  auto push_stores = [&]() {
    char* const pdst = pdst0 + (size_t)(call & 1u) * NR * BLK;
    asm volatile("global_store_dwordx4 %0, %1, off sc0 sc1\n\ts_nop 1" :: "v"(pdst), "v"(px0) : "memory");
    asm volatile("global_store_dwordx4 %0, %1, off offset:1024 sc0 sc1\n\ts_nop 1" :: "v"(pdst), "v"(px1) : "memory");
  };
  const int pj = (rank - 1 - dst) & 7;
  const int t_store = T_STORE(wid, pj);
  const int t_force = (t_store < 0 ? -1 : t_store) + 3;
  int pstate = pushes ? 0 : 2;
  if (pushes && t_store < 0) { push_stores(); pstate = 1; }
  STAMP(10);
  if (pstate == 1) { asm volatile("s_waitcnt vmcnt(0)" ::: "memory"); STAMP(11); if (lane == 0) __hip_atomic_store(pflag, call, __ATOMIC_RELAXED, __HIP_MEMORY_SCOPE_SYSTEM); pstate = 2; }
  int t_evt = pstate == 2 ? NSTEP : (pstate == 1 ? 0 : t_store);
  int t_mid = pstate == 1 ? 0 : NSTEP;
  auto push_event = [&](int t, bool force) {
    if (pstate == 0) { push_stores(); pstate = 1; if (wid == 0) t_mid = t; }
    else {
      unsigned sts; asm volatile("s_getreg_b32 %0, hwreg(HW_REG_IB_STS)" : "=s"(sts));
      if (force || (sts & 0x00C0000Fu) == 0u) {
        asm volatile("s_waitcnt vmcnt(0)" ::: "memory");
        if (lane == 0) __hip_atomic_store(pflag, call, __ATOMIC_RELAXED, __HIP_MEMORY_SCOPE_SYSTEM);
        pstate = 2; t_evt = NSTEP; t_mid = NSTEP;
      }
    }
  };

  const int wm = wid >> 1, wn = wid & 1;
  const int g = lane >> 4, l15 = lane & 15;
  const int fx = (l15 >> 1) & 7;
  const unsigned ax0 = (wm * 128 + l15) * 128 + ((g ^ fx) << 4);
  const unsigned ax1 = (wm * 128 + l15) * 128 + (((4 + g) ^ fx) << 4);
  const int q8 = l15 >> 1;
  unsigned aw[4];
#pragma unroll
  for (int j = 0; j < 4; ++j) aw[j] = XB + (16 * g + q8) * 128 + (((4 * wn + j) ^ (((q8 >> 1) & 3) | ((g & 1) << 2))) << 4) + 8 * (lane & 1);
  v4f acc[8][4];
#pragma unroll
  for (int i = 0; i < 8; ++i)
#pragma unroll
    for (int j = 0; j < 4; ++j) acc[i][j] = (v4f){};
  v8i alo[4], ahi[4], bf[4];
  const unsigned sb0 = (unsigned)(size_t)(LAS char*)smem;
#define RDA(dst, i, ii, sb) dst[i] = cat4(*(LAS v4i*)(size_t)(ax0 + (sb) + (ii) * 2048), *(LAS v4i*)(size_t)(ax1 + (sb) + (ii) * 2048))
#define TR(j, t, sb) __builtin_amdgcn_ds_read_tr8_b64_v2i32((LAS v2i*)(size_t)(aw[j] + (sb) + ((t) >> 1) * 8192 + ((t) & 1) * 1024))
#define RDB(j, sb) bf[j] = cat2(TR(j, 0, sb), TR(j, 1, sb), TR(j, 2, sb), TR(j, 3, sb))
#define MM(i, j, A, ai) acc[i][j] = __builtin_amdgcn_mfma_scale_f32_16x16x128_f8f6f4(A[ai], bf[j], acc[i][j], 1, 1, 0, 0, 0, 0)
#define MM4(i0, j, A) do { MM(i0, j, A, 0); MM(i0 + 1, j, A, 1); MM(i0 + 2, j, A, 2); MM(i0 + 3, j, A, 3); } while (0)
#define SGB(m, n) __builtin_amdgcn_sched_group_barrier(m, n, 0)
#define P1(n) SGB(0x8,1); SGB(0x100,n);
  WAIT_LGKM0();
  BAR();
  STAMP(2);
  call = __builtin_amdgcn_readfirstlane(lctl[1]);
  {
    const unsigned sb = sb0;
    RDA(alo, 0, 0, sb); RDA(alo, 1, 1, sb); RDA(alo, 2, 2, sb); RDA(alo, 3, 3, sb); RDB(0, sb); RDB(1, sb); RDB(2, sb);
  }
  unsigned sb = sb0;
  int slot = 0;
#pragma nounroll
  for (int t = 0; t < NSTEP; ++t) {
    const int nslot = (slot == 2) ? 0 : slot + 1;
    const unsigned sbn = sb0 + nslot * SLOT;
    SCHED();
    RDB(3, sb);
    MM(0, 0, alo, 0);
    P1(4)
    SCHED();
    if (t >= t_evt) push_event(t, t >= t_force);
    SCHED();
    RDA(ahi, 0, 4, sb); RDA(ahi, 1, 5, sb); RDA(ahi, 2, 6, sb); RDA(ahi, 3, 7, sb);
    MM(1, 0, alo, 1); MM(2, 0, alo, 2); MM(3, 0, alo, 3);
    MM4(0, 1, alo); MM4(0, 2, alo); MM4(0, 3, alo);
    P1(1) P1(1) P1(1) P1(1) P1(1) P1(1) P1(1) P1(1) SGB(0x8, 7);
    SCHED();
    WAIT_LGKM0();
    BAR();
    SCHED();
    RDA(alo, 0, 0, sbn);
    MM(4, 0, ahi, 0);
    P1(2)
    SCHED();
    if (t >= t_mid) push_event(t, false);
    SCHED();
    RDA(alo, 1, 1, sbn); RDA(alo, 2, 2, sbn); RDA(alo, 3, 3, sbn);
    MM(5, 0, ahi, 1); MM(6, 0, ahi, 2); MM(7, 0, ahi, 3);
    P1(2) P1(2) P1(2)
    SCHED();
    RDB(0, sbn);
    MM4(4, 1, ahi);
    P1(1) P1(1) P1(1) P1(1)
    SCHED();
    RDB(1, sbn);
    MM4(4, 2, ahi);
    P1(1) P1(1) P1(1) P1(1)
    SCHED();
    RDB(2, sbn);
    MM4(4, 3, ahi);
    P1(1) P1(1) P1(1) P1(1)
    SCHED();
    sb = sbn; slot = nslot;
  }
  STAMP(6);
  const unsigned tmo_seen = __hip_atomic_load(tmo, __ATOMIC_RELAXED, __HIP_MEMORY_SCOPE_AGENT);
  const bool bad = (tmo_seen == call) || (lctl[0] != 0u);
  const float s = bad ? __builtin_nanf("") : P.sx[0] * P.sw[0];
  __amdgpu_buffer_rsrc_t ro = __builtin_amdgcn_make_buffer_rsrc((void*)P.out, 0, MS * N * 4, 0x00020000);
  const int vo = ((m0 + wm * 128 + 8 * (lane >> 5)) * N + n0 + wn * 64 + (lane & 31)) * 4;
#pragma unroll
  for (int i = 0; i < 8; ++i)
#pragma unroll
    for (int r = 0; r < 4; ++r)
#pragma unroll
      for (int jp = 0; jp < 2; ++jp) {
        unsigned o[2];
#pragma unroll
        for (int jj = 0; jj < 2; ++jj) {
          const float y = acc[i][2 * jp + jj][r] * s;
          const float c = __builtin_amdgcn_fmed3f(y, -60.f, 60.f);
          const float e = __builtin_amdgcn_exp2f(c * -1.44269504088896f);
          o[jj] = __builtin_bit_cast(unsigned, y * __builtin_amdgcn_rcpf(1.0f + e));
        }
        const auto sw2 = __builtin_amdgcn_permlane16_swap(o[0], o[1], false, false);
        __builtin_amdgcn_raw_buffer_store_b32(sw2[0], ro, vo + jp * 128, (16 * i + r) * N * 4, 16);
        __builtin_amdgcn_raw_buffer_store_b32(sw2[1], ro, vo + jp * 128, (16 * i + r + 4) * N * 4, 16);
      }
}

extern "C" void kernel_launch(int rank, int nranks, ncclComm_t comm,
                              hipStream_t stream,
                              void* const* d_in, const int* in_sizes, int n_in,
                              void* d_out, int out_size,
                              const int* in_dtypes, int out_dtype,
                              void* const* d_out_all,
                              void* const* d_ws_all, size_t ws_bytes) {
  (void)comm; (void)in_dtypes; (void)out_dtype; (void)d_out_all;
  if (nranks != NR || rank < 0 || rank >= NR || n_in != 4 || in_sizes[0] != NR * BLK || in_sizes[1] != K * N ||
      out_size != MS * N || ws_bytes < WS_NEED) {
    fprintf(stderr, "kernel_launch: unexpected problem shape\n");
    return;
  }
  static bool attr_set = false;
  if (!attr_set) {
    HIP_CHECK(hipFuncSetAttribute((const void*)a2a_gemm_k, hipFuncAttributeMaxDynamicSharedMemorySize, LDS_BYTES));
    attr_set = true;
  }
  Params P;
  P.x = (const char*)d_in[0]; P.w = (const char*)d_in[1]; P.sx = (const float*)d_in[2]; P.sw = (const float*)d_in[3];
  P.out = (float*)d_out; P.myws = (char*)d_ws_all[rank];
  for (int p = 0; p < NR; ++p) P.ws[p] = (char*)d_ws_all[p];
  P.rank = rank; P.pad = 0;
  a2a_gemm_k<<<256, 512, LDS_BYTES, stream>>>(P);
  HIP_CHECK(hipGetLastError());
}
```

```cpp
#include <hip/hip_runtime.h>
#include <rccl/rccl.h>
#include <cstdio>

#define HIP_CHECK(expr) do { hipError_t _e = (expr); if (_e != hipSuccess) { \
    fprintf(stderr, "HIP error %s at %s:%d\n", hipGetErrorString(_e), __FILE__, __LINE__); \
    return; } } while (0)

#define LAS __attribute__((address_space(3)))
typedef int v2i __attribute__((ext_vector_type(2)));
typedef int v4i __attribute__((ext_vector_type(4)));
typedef int v8i __attribute__((ext_vector_type(8)));
typedef unsigned v4u __attribute__((ext_vector_type(4)));
typedef float v2f __attribute__((ext_vector_type(2)));
typedef float v4f __attribute__((ext_vector_type(4)));

constexpr int NR = 8;
constexpr int MS = 512, N = 16384, K = 4096;
constexpr int KS = K / NR;
constexpr int BLK = MS * KS;
constexpr int SLOT = 49152, XB = 32768;
constexpr int NSLOT = 3, NSTEP = 32, NREM = 28;
constexpr int LDS_CTRL = NSLOT * SLOT;
constexpr int LDS_FLG = LDS_CTRL + 256;
constexpr int LDS_SCRAP = LDS_FLG + 2048 + 256;
constexpr int LDS_BYTES = LDS_SCRAP + 256;
constexpr size_t WS_CTR = 0;
constexpr size_t WS_TMO = 32768;
constexpr size_t WS_FLAG = 36864;
constexpr size_t WS_X = 1 << 20;
constexpr size_t WS_NEED = WS_X + 2 * (size_t)NR * BLK;
constexpr unsigned long long DEADLINE_TICKS = 100000000ull;

#define POLL_LOW 5
#define POLL_EVERY 3
#define POLL_URGENT 3
#define FIRST_LOOK 0
#define T_STORE(w, j) ((w) == 0 ? -1 : 2 * ((w) - 1))
#define SLOT2_EARLY 0
#define OUT_AUX 16
#define PF_ON 1
#define PF_HELP 0
#define PFA_N0 1
#define PF_T2 2

struct Params {
  const char* x; const char* w; const float* sx; const float* sw; float* out; char* myws;
  char* ws[NR];
  int rank; int pad;
};

#define LTRACE(i) do {} while (0)
#define STAMP(i) do {} while (0)
#define STAMP_FLUSH(n) do {} while (0)

__device__ __forceinline__ v8i cat4(v4i lo, v4i hi) { return __builtin_shufflevector(lo, hi, 0, 1, 2, 3, 4, 5, 6, 7); }
__device__ __forceinline__ v8i cat2(v2i a, v2i b, v2i c, v2i d) {
  v4i lo = __builtin_shufflevector(a, b, 0, 1, 2, 3), hi = __builtin_shufflevector(c, d, 0, 1, 2, 3);
  return cat4(lo, hi);
}
#define WAIT_LGKM0() __builtin_amdgcn_s_waitcnt(0xC07F)
#define BAR() __builtin_amdgcn_s_barrier()
#define SCHED() __builtin_amdgcn_sched_barrier(0)

__device__ __forceinline__ void piece_pq(int u, int rank, int& p, int& q) { q = u / 7; p = (rank + 1 + (u - q * 7)) & 7; }

__global__ __launch_bounds__(512, 2) void a2a_gemm_k(const Params P) {
  extern __shared__ __attribute__((aligned(16))) char smem[];
  const int tid = threadIdx.x, wid = __builtin_amdgcn_readfirstlane(tid >> 6), lane = tid & 63;
  const int b = blockIdx.x, grp = b & 7, idx = b >> 3;
  const int mh = idx & 1, nt = grp * 16 + (idx >> 1);
  const int m0 = mh * 256, n0 = nt * 128;
  const int rank = P.rank;
  char* const myws = P.myws;
  STAMP(0);
  unsigned* const tmo = (unsigned*)(myws + WS_TMO);
  volatile LAS unsigned* const lctl = (volatile LAS unsigned*)((LAS char*)smem + LDS_CTRL);

  if (wid >= 4) {
    const int lw = wid - 4;
    __amdgpu_buffer_rsrc_t rxl = __builtin_amdgcn_make_buffer_rsrc((void*)(P.x + (size_t)rank * BLK), 0, BLK, 0x00020000);
    __amdgpu_buffer_rsrc_t rw = __builtin_amdgcn_make_buffer_rsrc((void*)P.w, 0, K * N, 0x00020000);
    const int r8 = lane >> 3, sl = lane & 7;
    const int vx0 = (m0 + lw * 64 + r8) * KS + ((sl ^ (r8 >> 1)) << 4);
    const int vx1 = (m0 + lw * 64 + r8) * KS + ((sl ^ (4 + (r8 >> 1))) << 4);
    const int vs0 = (m0 + lw * 64 + r8) * 128 + ((sl ^ (r8 >> 1)) << 4);
    const int vs1 = (m0 + lw * 64 + r8) * 128 + ((sl ^ (4 + (r8 >> 1))) << 4);
    const int vw0 = (lw * 32 + r8) * N + n0 + ((sl ^ ((r8 >> 1) & 3)) << 4);
    const int vw1 = (lw * 32 + r8) * N + n0 + ((sl ^ (((r8 >> 1) & 3) | 4)) << 4);
    auto issue_w = [&](int p, int q, int slot) {
      const int sow = (p * KS + q * 128) * N;
      LAS char* lwp = (LAS char*)smem + slot * SLOT + XB + lw * 32 * 128;
#pragma unroll
      for (int i = 0; i < 4; ++i)
        __builtin_amdgcn_raw_ptr_buffer_load_lds(rw, lwp + i * 1024, 16, (i & 2) ? vw1 : vw0, sow + i * 8 * N, 0, 0);
    };
    auto issue_local = [&](int s, int slot) {
      issue_w(rank, s, slot);
      LAS char* lx = (LAS char*)smem + slot * SLOT + lw * 64 * 128;
#pragma unroll
      for (int i = 0; i < 8; ++i)
        __builtin_amdgcn_raw_ptr_buffer_load_lds(rxl, lx + i * 1024, 16, (i & 1) ? vx1 : vx0, s * 128 + i * 8 * KS, 0, 0);
    };
    const int slow_j = (3 - (int)((unsigned)(size_t)P.w >> 7)) & 7;
    const int pfm = (!PF_ON || lw != 3) ? 0 : (((idx >> 1) & 7) == slow_j ? (PF_HELP ? 0 : 1) : (PF_HELP && (((idx >> 1) ^ 1) & 7) == slow_j ? 2 : 1));
    const bool pfw = pfm > 0;
    const int pf_voff = (4 * idx + (lane >> 4)) * N + grp * 2048 + (lane & 15) * 128;
    const int pf_voff2 = (4 * (idx ^ 2) + (lane >> 4)) * N + grp * 2048 + (lane & 15) * 128;
    int pf_u = 0;
    auto prefetch = [&]() {
      int p, q; piece_pq(pf_u < NREM ? pf_u : NREM - 1, rank, p, q);
      __builtin_amdgcn_raw_ptr_buffer_load_lds(rw, (LAS char*)smem + LDS_SCRAP, 4, pf_voff, (p * KS + q * 128) * N, 0, 0);
      if (pfm == 2) __builtin_amdgcn_raw_ptr_buffer_load_lds(rw, (LAS char*)smem + LDS_SCRAP, 4, pf_voff2, (p * KS + q * 128) * N, 0, 0);
      ++pf_u;
    };
    auto prefetch_step = [&](int t) { prefetch(); if (t <= PF_T2) prefetch(); };
    auto wait_slot = [&](int t) {
      const int c = pfm * ((t >= PF_T2 + 3 || t == 1) ? 2 : (t == PF_T2 + 2 ? 3 : 4));
      if (c == 0) asm volatile("s_waitcnt vmcnt(12)" ::: "memory");
      else if (c == 2) asm volatile("s_waitcnt vmcnt(14)" ::: "memory");
      else if (c == 3) asm volatile("s_waitcnt vmcnt(15)" ::: "memory");
      else if (c == 4) asm volatile("s_waitcnt vmcnt(16)" ::: "memory");
      else if (c == 6) asm volatile("s_waitcnt vmcnt(18)" ::: "memory");
      else asm volatile("s_waitcnt vmcnt(20)" ::: "memory");
    };
    auto wait_first = [&]() {
      if (pfm == 2) asm volatile("s_waitcnt vmcnt(%0)" :: "n"(12 + 2 * PFA_N0) : "memory");
      else if (pfm == 1) asm volatile("s_waitcnt vmcnt(%0)" :: "n"(12 + PFA_N0) : "memory");
      else asm volatile("s_waitcnt vmcnt(12)" ::: "memory");
    };
    issue_local(0, 0); issue_local(1, 1);
    if (pfw) {
#pragma unroll
      for (int u = 0; u < PFA_N0; ++u) prefetch();
    }
    if (lw == 0 && lane < 40 && lane != 1) lctl[lane] = 0u;
    wait_first();
    asm volatile("s_waitcnt lgkmcnt(0)" ::: "memory");
    BAR();
    issue_local(2, 2);
    LTRACE(32);
    STAMP(1);
    const unsigned call = __builtin_amdgcn_readfirstlane(lctl[1]);
    __amdgpu_buffer_rsrc_t rxs = __builtin_amdgcn_make_buffer_rsrc((void*)(myws + WS_X + (size_t)(call & 1u) * NR * BLK), 0, NR * BLK, 0x00020000);
    __amdgpu_buffer_rsrc_t rfl = __builtin_amdgcn_make_buffer_rsrc((void*)(myws + WS_FLAG + mh * 2048), 0, 2048, 0x00020000);
    unsigned flg_addr, my_entry;
    {
      int p, q; piece_pq(min(lane >> 1, NREM - 1), rank, p, q);
      flg_addr = (unsigned)(size_t)((LAS char*)smem + LDS_FLG) + (p * 4 + q) * 64 + (lane & 1) * 32;
      my_entry = 0x100u | (unsigned)(p * 4 + q);
    }
    auto poll_issue = [&]() {
      __builtin_amdgcn_raw_ptr_buffer_load_lds(rfl, (LAS char*)smem + LDS_FLG, 16, lane * 16, 0, 0, 17);
      __builtin_amdgcn_raw_ptr_buffer_load_lds(rfl, (LAS char*)smem + LDS_FLG + 1024, 16, lane * 16, 1024, 0, 17);
    };
    auto poll_eval = [&]() -> unsigned long long {
      const v4u f0 = *(volatile LAS v4u*)(size_t)flg_addr, f1 = *(volatile LAS v4u*)(size_t)(flg_addr + 16);
      const unsigned m = min(min(min(f0.x, f0.y), min(f0.z, f0.w)), min(min(f1.x, f1.y), min(f1.z, f1.w)));
      const unsigned long long ok = __ballot(m >= call && lane < 2 * NREM);
      return ok & (ok >> 1) & 0x5555555555555555ull;
    };
    unsigned long long ready = 0, used = 0;
    constexpr unsigned long long ALL = 0x0055555555555555ull;
    int nsched = 4;
    auto schedule = [&](int upto) {
      const unsigned long long avail = ready & ~used;
      const int q0 = (__builtin_ctzll(ALL & ~used) >> 1) / 7;
      const unsigned long long gq = 0x1555ull << (14 * q0);
      const unsigned long long first = ready == ALL ? avail : (avail & gq), rest = avail & ~first;
      const int nfirst = __builtin_popcountll(first);
      const bool in_first = (first >> lane) & 1ull;
      const unsigned long long m = in_first ? first : rest;
      const int pos = nsched + (in_first ? 0 : nfirst) + (int)__builtin_amdgcn_mbcnt_hi((unsigned)(m >> 32), __builtin_amdgcn_mbcnt_lo((unsigned)m, 0u));
      const bool take = ((avail >> lane) & 1ull) && (in_first || pos <= upto);
      if (take) lctl[4 + pos] = my_entry;
      const unsigned long long taken = __ballot(take);
      used |= taken;
      nsched += __builtin_popcountll(taken);
    };
    auto issue_remote = [&](unsigned e, int slot) {
      const int pq = (int)(e & 31u);
      LAS char* lwp = (LAS char*)smem + slot * SLOT + XB + lw * 32 * 128;
      const int sow = pq * 128 * N;
#pragma unroll
      for (int i = 0; i < 4; ++i)
        __builtin_amdgcn_raw_ptr_buffer_load_lds(rw, lwp + i * 1024, 16, (i & 2) ? vw1 : vw0, sow + i * 8 * N, 0, 0);
      LAS char* lx = (LAS char*)smem + slot * SLOT + lw * 64 * 128;
      const int sox = pq * (MS * 128);
#pragma unroll
      for (int i = 0; i < 8; ++i)
        __builtin_amdgcn_raw_ptr_buffer_load_lds(rxs, lx + i * 1024, 16, (i & 1) ? vs1 : vs0, sox + i * 1024, 0, 17);
    };
    wait_first();
    BAR();
    LTRACE(0);
    int poll_age = 0;
    if (FIRST_LOOK && lw == 0 && ready != ALL) { poll_issue(); poll_age = 3; }
    issue_local(3, 0);
    if (pfw) prefetch_step(0);
    int slot = 1;
    int t = 1;
    unsigned tbl = 0;
    for (; t < NSTEP - 3; ++t) {
      if (poll_age == 1 || poll_age == 2) asm volatile("s_waitcnt vmcnt(14)" ::: "memory");
      else wait_slot(t);
      const int s = t + 3;
      if (lw == 0) {
        if (poll_age == 2) {
          unsigned sts; asm volatile("s_getreg_b32 %0, hwreg(HW_REG_IB_STS)" : "=s"(sts));
          if (((sts & 15u) | ((sts >> 18) & 0x30u)) <= 12u) { asm volatile("s_waitcnt vmcnt(12)" ::: "memory"); poll_age = 3; }
        }
        bool news = false;
        if (poll_age == 3) { ready |= poll_eval(); poll_age = 0; news = true; }
        if ((news || nsched <= s + 2) && (ready & ~used)) schedule(s + 2);
      }
      unsigned e = __builtin_amdgcn_readlane(tbl, s);
      BAR();
      LTRACE(t);
      if (!e) {
        tbl = lctl[4 + (lane & 31)];
        e = __builtin_amdgcn_readlane(tbl, s);
        if (!e) {
          if (lw == 0) {
            const unsigned long long t0 = __builtin_amdgcn_s_memrealtime();
            for (unsigned n = 1;; ++n) {
              poll_issue();
              asm volatile("s_waitcnt vmcnt(0)" ::: "memory");
              ready |= poll_eval();
              if (ready & ~used) break;
              if ((n & 15u) == 0u) {
                const unsigned tv = __builtin_amdgcn_readfirstlane(__hip_atomic_load(tmo, __ATOMIC_RELAXED, __HIP_MEMORY_SCOPE_AGENT));
                if (tv == call || __builtin_amdgcn_s_memrealtime() - t0 > DEADLINE_TICKS) {
                  if (lane == 0) { __hip_atomic_store(tmo, call, __ATOMIC_RELAXED, __HIP_MEMORY_SCOPE_AGENT); lctl[0] = 1u; }
                  ready = ALL;
                  break;
                }
              }
            }
            poll_age = 0;
            schedule(s + 2);
          }
          for (;;) {
            tbl = lctl[4 + (lane & 31)];
            e = __builtin_amdgcn_readlane(tbl, s);
            if (e) break;
            __builtin_amdgcn_s_sleep(1);
          }
        }
      }
      const unsigned tbl_next = lctl[4 + (lane & 31)];
      const int have = (lw == 0 && ready != ALL && poll_age == 0) ? nsched - s - 1 + __builtin_popcountll(ready & ~used) : 99;
      if (have < POLL_URGENT) poll_issue();
      issue_remote(e, slot);
      if (pfw) prefetch_step(t);
      if (poll_age) ++poll_age;
      else if (have < POLL_URGENT) poll_age = 3;
      else if (have < POLL_LOW || (have < 99 && (t & POLL_EVERY) == 0)) { poll_issue(); poll_age = 1; }
      tbl = tbl_next;
      slot = (slot == 2) ? 0 : slot + 1;
      if (__builtin_amdgcn_readlane(tbl, NSTEP - 1) != 0u && poll_age == 0) { ++t; break; }
    }
    for (; t < NSTEP - 3; ++t) {
      wait_slot(t);
      const unsigned e = __builtin_amdgcn_readlane(tbl, t + 3);
      BAR();
      LTRACE(t);
      issue_remote(e, slot);
      if (pfw) prefetch_step(t);
      slot = (slot == 2) ? 0 : slot + 1;
    }
    if (lw == 0 && lane == 0) {
      __amdgpu_buffer_rsrc_t rt = __builtin_amdgcn_make_buffer_rsrc((void*)tmo, 0, 4, 0x00020000);
      __builtin_amdgcn_raw_ptr_buffer_load_lds(rt, (LAS char*)smem + LDS_CTRL + 8, 4, 0, 0, 0, 16);
    }
    wait_slot(29);
    BAR();
    LTRACE(29);
    asm volatile("s_waitcnt vmcnt(0)" ::: "memory");
    BAR();
    LTRACE(30);
    BAR();
    LTRACE(31);
    STAMP_FLUSH(12);
    return;
  }

  const int dst = (grp + idx) & 7;
  const bool pushes = (dst != rank);
  v4i px0, px1;
  const int poff = (idx * 16 + (lane >> 3)) * KS + wid * 128 + (lane & 7) * 16;
  char* const dws = P.ws[dst];
  const float sx0 = P.sx[0], sw0 = P.sw[0];
  unsigned call;
  {
    const char* src = P.x + (size_t)dst * BLK + poff;
    if (wid == 0) {
      unsigned* cp = (unsigned*)(myws + WS_CTR) + b * 32;
      unsigned tick = 0, one = 1u;
      asm volatile("s_mov_b64 exec, 1\n\tglobal_atomic_add %0, %3, %4, off sc0\n\ts_mov_b64 exec, -1\n\t"
                   "global_load_dwordx4 %1, %5, off\n\tglobal_load_dwordx4 %2, %6, off\n\ts_waitcnt vmcnt(0)"
                   : "+v"(tick), "=&v"(px0), "=&v"(px1) : "v"(cp), "v"(one), "v"(src), "v"(src + 8 * KS) : "memory");
      call = __builtin_amdgcn_readfirstlane(tick) + 1u;
      if (lane == 0) lctl[1] = call;
    } else {
      asm volatile("global_load_dwordx4 %0, %2, off\n\tglobal_load_dwordx4 %1, %3, off\n\ts_waitcnt vmcnt(0)"
                   : "=&v"(px0), "=&v"(px1) : "v"(src), "v"(src + 8 * KS) : "memory");
      call = 0;
    }
  }
  const float scale = __builtin_bit_cast(float, __builtin_amdgcn_readfirstlane(__builtin_bit_cast(int, sx0 * sw0)));
  STAMP(1);
  char* const pdst0 = dws + WS_X + (size_t)rank * BLK + wid * (MS * 128) + (idx * 16 + (lane >> 3)) * 128 + (lane & 7) * 16;
  unsigned* const pflag = (unsigned*)(dws + WS_FLAG) + (((idx >> 4) * 8 + rank) * 4 + wid) * 16 + (idx & 15);
  auto push_stores = [&]() {
    char* const pdst = pdst0 + (size_t)(call & 1u) * NR * BLK;
    asm volatile("global_store_dwordx4 %0, %1, off sc0 sc1\n\ts_nop 1" :: "v"(pdst), "v"(px0) : "memory");
    asm volatile("global_store_dwordx4 %0, %1, off offset:1024 sc0 sc1\n\ts_nop 1" :: "v"(pdst), "v"(px1) : "memory");
  };
  const int pj = (rank - 1 - dst) & 7;
  const int t_store = T_STORE(wid, pj);
  const int t_force = (t_store < 0 ? -1 : t_store) + 3;
  int pstate = pushes ? 0 : 2;
  if (pushes && t_store < 0) { push_stores(); pstate = 1; }
  STAMP(10);
  if (pstate == 1) { asm volatile("s_waitcnt vmcnt(0)" ::: "memory"); STAMP(11); if (lane == 0) __hip_atomic_store(pflag, call, __ATOMIC_RELAXED, __HIP_MEMORY_SCOPE_SYSTEM); pstate = 2; }
  int t_evt = pstate == 2 ? NSTEP : (pstate == 1 ? 0 : t_store);
  int t_mid = pstate == 1 ? 0 : NSTEP;
  auto push_event = [&](int t, bool force) {
    if (pstate == 0) { push_stores(); pstate = 1; if (wid == 0) t_mid = t; }
    else {
      unsigned sts; asm volatile("s_getreg_b32 %0, hwreg(HW_REG_IB_STS)" : "=s"(sts));
      if (force || (sts & 0x00C0000Fu) == 0u) {
        asm volatile("s_waitcnt vmcnt(0)" ::: "memory");
        if (lane == 0) __hip_atomic_store(pflag, call, __ATOMIC_RELAXED, __HIP_MEMORY_SCOPE_SYSTEM);
        pstate = 2; t_evt = NSTEP; t_mid = NSTEP;
      }
    }
  };

  const int wm = wid >> 1, wn = wid & 1;
  const int g = lane >> 4, l15 = lane & 15;
  const int fx = (l15 >> 1) & 7;
  const unsigned ax0 = (wm * 128 + l15) * 128 + ((g ^ fx) << 4);
  const unsigned ax1 = (wm * 128 + l15) * 128 + (((4 + g) ^ fx) << 4);
  const int q8 = l15 >> 1;
  unsigned aw[4];
#pragma unroll
  for (int j = 0; j < 4; ++j) aw[j] = XB + (16 * g + q8) * 128 + (((4 * wn + j) ^ (((q8 >> 1) & 3) | ((g & 1) << 2))) << 4) + 8 * (lane & 1);
  v4f acc[8][4];
#pragma unroll
  for (int i = 0; i < 8; ++i)
#pragma unroll
    for (int j = 0; j < 4; ++j) acc[i][j] = (v4f){};
  v8i alo[4], ahi[4], bf[4];
  const unsigned sb0 = (unsigned)(size_t)(LAS char*)smem;
#define RDA(dst, i, ii, sb) dst[i] = cat4(*(LAS v4i*)(size_t)(ax0 + (sb) + (ii) * 2048), *(LAS v4i*)(size_t)(ax1 + (sb) + (ii) * 2048))
#define TR(j, t, sb) __builtin_amdgcn_ds_read_tr8_b64_v2i32((LAS v2i*)(size_t)(aw[j] + (sb) + ((t) >> 1) * 8192 + ((t) & 1) * 1024))
#define RDB(j, sb) bf[j] = cat2(TR(j, 0, sb), TR(j, 1, sb), TR(j, 2, sb), TR(j, 3, sb))
#define MM(i, j, A, ai) acc[i][j] = __builtin_amdgcn_mfma_scale_f32_16x16x128_f8f6f4(A[ai], bf[j], acc[i][j], 1, 1, 0, 0, 0, 0)
#define MM4(i0, j, A) do { MM(i0, j, A, 0); MM(i0 + 1, j, A, 1); MM(i0 + 2, j, A, 2); MM(i0 + 3, j, A, 3); } while (0)
#define SGB(m, n) __builtin_amdgcn_sched_group_barrier(m, n, 0)
#define P1(n) SGB(0x8,1); SGB(0x100,n);
  WAIT_LGKM0();
  BAR();
  STAMP(2);
  call = __builtin_amdgcn_readfirstlane(lctl[1]);
  {
    const unsigned sb = sb0;
    RDA(alo, 0, 0, sb); RDA(alo, 1, 1, sb); RDA(alo, 2, 2, sb); RDA(alo, 3, 3, sb); RDB(0, sb); RDB(1, sb); RDB(2, sb);
  }
  unsigned sb = sb0;
  int slot = 0;
#pragma nounroll
  for (int t = 0; t < NSTEP; ++t) {
    const int nslot = (slot == 2) ? 0 : slot + 1;
    const unsigned sbn = sb0 + nslot * SLOT;
    SCHED();
    RDB(3, sb);
    MM(0, 0, alo, 0);
    P1(4)
    SCHED();
    if (t >= t_evt) push_event(t, t >= t_force);
    SCHED();
    RDA(ahi, 0, 4, sb); RDA(ahi, 1, 5, sb); RDA(ahi, 2, 6, sb); RDA(ahi, 3, 7, sb);
    MM(1, 0, alo, 1); MM(2, 0, alo, 2); MM(3, 0, alo, 3);
    MM4(0, 1, alo); MM4(0, 2, alo); MM4(0, 3, alo);
    P1(1) P1(1) P1(1) P1(1) P1(1) P1(1) P1(1) P1(1) SGB(0x8, 7);
    SCHED();
    WAIT_LGKM0();
    BAR();
    SCHED();
    RDA(alo, 0, 0, sbn);
    MM(4, 0, ahi, 0);
    P1(2)
    SCHED();
    if (t >= t_mid) push_event(t, false);
    SCHED();
    RDA(alo, 1, 1, sbn); RDA(alo, 2, 2, sbn); RDA(alo, 3, 3, sbn);
    MM(5, 0, ahi, 1); MM(6, 0, ahi, 2); MM(7, 0, ahi, 3);
    P1(2) P1(2) P1(2)
    SCHED();
    RDB(0, sbn);
    MM4(4, 1, ahi);
    P1(1) P1(1) P1(1) P1(1)
    SCHED();
    RDB(1, sbn);
    MM4(4, 2, ahi);
    P1(1) P1(1) P1(1) P1(1)
    SCHED();
    RDB(2, sbn);
    MM4(4, 3, ahi);
    P1(1) P1(1) P1(1) P1(1)
    SCHED();
    sb = sbn; slot = nslot;
  }
  STAMP(6);
  const bool bad = (lctl[2] == call) || (lctl[0] != 0u);
  const float s = bad ? __builtin_nanf("") : scale;
  constexpr float LOG2E = 1.44269504088896f, ZMAX = 60.0f * LOG2E;
  const v2f s2 = {s, s}, c2 = {-s * LOG2E, -s * LOG2E};
  __amdgpu_buffer_rsrc_t ro = __builtin_amdgcn_make_buffer_rsrc((void*)P.out, 0, MS * N * 4, 0x00020000);
  const int vo = ((m0 + wm * 128 + 8 * (lane >> 5)) * N + n0 + wn * 64 + (lane & 31)) * 4;
  auto silu2 = [&](float a0, float a1) -> v2f {
    const v2f a = {a0, a1};
    const v2f y = a * s2, z = a * c2;
    const v2f d = (v2f){__builtin_amdgcn_exp2f(__builtin_amdgcn_fmed3f(z.x, -ZMAX, ZMAX)), __builtin_amdgcn_exp2f(__builtin_amdgcn_fmed3f(z.y, -ZMAX, ZMAX))} + 1.0f;
    return y * (v2f){__builtin_amdgcn_rcpf(d.x), __builtin_amdgcn_rcpf(d.y)};
  };
#pragma unroll
  for (int i = 0; i < 8; ++i)
#pragma unroll
    for (int rp = 0; rp < 4; rp += 2)
#pragma unroll
      for (int jp = 0; jp < 2; ++jp) {
        const v2f u = silu2(acc[i][2 * jp][rp], acc[i][2 * jp][rp + 1]), v = silu2(acc[i][2 * jp + 1][rp], acc[i][2 * jp + 1][rp + 1]);
        const float u0 = u.x, u1 = u.y, v0 = v.x, v1 = v.y;
        const auto w0 = __builtin_amdgcn_permlane16_swap(__float_as_uint(u0), __float_as_uint(v0), false, false);
        const auto w1 = __builtin_amdgcn_permlane16_swap(__float_as_uint(u1), __float_as_uint(v1), false, false);
        __builtin_amdgcn_raw_buffer_store_b32(w0[0], ro, vo + jp * 128, (16 * i + rp) * N * 4, OUT_AUX);
        __builtin_amdgcn_raw_buffer_store_b32(w0[1], ro, vo + jp * 128, (16 * i + rp + 4) * N * 4, OUT_AUX);
        __builtin_amdgcn_raw_buffer_store_b32(w1[0], ro, vo + jp * 128, (16 * i + rp + 1) * N * 4, OUT_AUX);
        __builtin_amdgcn_raw_buffer_store_b32(w1[1], ro, vo + jp * 128, (16 * i + rp + 5) * N * 4, OUT_AUX);
      }
}

extern "C" void kernel_launch(int rank, int nranks, ncclComm_t comm,
                              hipStream_t stream,
                              void* const* d_in, const int* in_sizes, int n_in,
                              void* d_out, int out_size,
                              const int* in_dtypes, int out_dtype,
                              void* const* d_out_all,
                              void* const* d_ws_all, size_t ws_bytes) {
  (void)comm; (void)in_dtypes; (void)out_dtype; (void)d_out_all;
  if (nranks != NR || rank < 0 || rank >= NR || n_in != 4 || in_sizes[0] != NR * BLK || in_sizes[1] != K * N ||
      out_size != MS * N || ws_bytes < WS_NEED) {
    fprintf(stderr, "kernel_launch: unexpected problem shape\n");
    return;
  }
  static bool attr_set = false;
  if (!attr_set) {
    HIP_CHECK(hipFuncSetAttribute((const void*)a2a_gemm_k, hipFuncAttributeMaxDynamicSharedMemorySize, LDS_BYTES));
    attr_set = true;
  }
  Params P;
  P.x = (const char*)d_in[0]; P.w = (const char*)d_in[1]; P.sx = (const float*)d_in[2]; P.sw = (const float*)d_in[3];
  P.out = (float*)d_out; P.myws = (char*)d_ws_all[rank];
  for (int p = 0; p < NR; ++p) P.ws[p] = (char*)d_ws_all[p];
  P.rank = rank; P.pad = 0;
  a2a_gemm_k<<<256, 512, LDS_BYTES, stream>>>(P);
  HIP_CHECK(hipGetLastError());
}
```

```cpp
#include <hip/hip_runtime.h>
#include <rccl/rccl.h>
#include <cstdio>

#define HIP_CHECK(expr) do { hipError_t _e = (expr); if (_e != hipSuccess) { \
    fprintf(stderr, "HIP error %s at %s:%d\n", hipGetErrorString(_e), __FILE__, __LINE__); \
    return; } } while (0)

#define LAS __attribute__((address_space(3)))
typedef int v2i __attribute__((ext_vector_type(2)));
typedef int v4i __attribute__((ext_vector_type(4)));
typedef int v8i __attribute__((ext_vector_type(8)));
typedef unsigned v4u __attribute__((ext_vector_type(4)));
typedef float v2f __attribute__((ext_vector_type(2)));
typedef float v4f __attribute__((ext_vector_type(4)));

constexpr int NR = 8;
constexpr int MS = 512, N = 16384, K = 4096;
constexpr int KS = K / NR;
constexpr int BLK = MS * KS;
constexpr int SLOT = 49152, XB = 16384;
constexpr int NSLOT = 3, NSTEP = 32, NREM = 28;
constexpr int LDS_CTRL = NSLOT * SLOT;
constexpr int LDS_FLG = LDS_CTRL + 256;
constexpr int LDS_SCRAP = LDS_FLG + 2048 + 256;
constexpr int LDS_BYTES = LDS_SCRAP + 256;
constexpr size_t WS_CTR = 0;
constexpr size_t WS_TMO = 32768;
constexpr size_t WS_FLAG = 36864;
constexpr size_t WS_X = 1 << 20;
constexpr size_t WS_NEED = WS_X + 2 * (size_t)NR * BLK;
constexpr unsigned long long DEADLINE_TICKS = 150000000ull;

#define POLL_LOW 4
#define POLL_URGENT 3
#define POLL_EVERY 7
#define T_STORE(w) ((w) == 0 ? -1 : 2 * ((w) - 1))
#define OUT_AUX 16
#define PF_ON 1
#define PF_T2 1

struct Params {
  const char* x; const char* w; const float* sx; const float* sw; float* out; char* myws;
  char* ws[NR];
  int rank; int pad;
};

__device__ __forceinline__ v8i cat4(v4i lo, v4i hi) { return __builtin_shufflevector(lo, hi, 0, 1, 2, 3, 4, 5, 6, 7); }
__device__ __forceinline__ v8i cat2(v2i a, v2i b, v2i c, v2i d) {
  v4i lo = __builtin_shufflevector(a, b, 0, 1, 2, 3), hi = __builtin_shufflevector(c, d, 0, 1, 2, 3);
  return cat4(lo, hi);
}
#define WAIT_LGKM0() __builtin_amdgcn_s_waitcnt(0xC07F)
#define BAR() __builtin_amdgcn_s_barrier()
#define SCHED() __builtin_amdgcn_sched_barrier(0)

__device__ __forceinline__ void piece_pq(int u, int rank, int& p, int& q) { q = u / 7; p = (rank + 1 + (u - q * 7)) & 7; }

__global__ __launch_bounds__(512, 2) void a2a_gemm_k(const Params P) {
  extern __shared__ __attribute__((aligned(16))) char smem[];
  const int tid = threadIdx.x, wid = __builtin_amdgcn_readfirstlane(tid >> 6), lane = tid & 63;
  const int b = blockIdx.x, grp = b & 7, idx = b >> 3;
  const int mq = idx & 3, nt = grp * 8 + (idx >> 2);
  const int m0 = mq * 128, n0 = nt * 256;
  const int rank = P.rank;
  char* const myws = P.myws;
  unsigned* const tmo = (unsigned*)(myws + WS_TMO);
  volatile LAS unsigned* const lctl = (volatile LAS unsigned*)((LAS char*)smem + LDS_CTRL);
  volatile LAS unsigned* const lsch = lctl + 4;

  if (wid >= 4) {
    const int lw = wid - 4;
    __amdgpu_buffer_rsrc_t rxl = __builtin_amdgcn_make_buffer_rsrc((void*)(P.x + (size_t)rank * BLK), 0, BLK, 0x00020000);
    __amdgpu_buffer_rsrc_t rw = __builtin_amdgcn_make_buffer_rsrc((void*)P.w, 0, K * N, 0x00020000);
    const int r8 = lane >> 3, sl = lane & 7;
    const int vx0 = (m0 + lw * 32 + r8) * KS + ((sl ^ (r8 >> 1)) << 4);
    const int vx1 = (m0 + lw * 32 + r8) * KS + ((sl ^ (4 + (r8 >> 1))) << 4);
    const int vs0 = (m0 + lw * 32 + r8) * 128 + ((sl ^ (r8 >> 1)) << 4);
    const int vs1 = (m0 + lw * 32 + r8) * 128 + ((sl ^ (4 + (r8 >> 1))) << 4);
    const int r4 = lane >> 4, s16 = lane & 15;
    int vw[4];
#pragma unroll
    for (int v = 0; v < 4; ++v) vw[v] = (lw * 32 + r4) * N + n0 + ((s16 ^ ((4 * (v & 1) + r4) | ((v >> 1) << 3))) << 4);
    auto issue_w = [&](int sow, int slot) {
      LAS char* lwp = (LAS char*)smem + slot * SLOT + XB + lw * 32 * 256;
#pragma unroll
      for (int i = 0; i < 8; ++i)
        __builtin_amdgcn_raw_ptr_buffer_load_lds(rw, lwp + i * 1024, 16, vw[(i & 1) | ((i >> 2) << 1)], sow + i * 4 * N, 0, 0);
    };
    auto issue_local = [&](int s, int slot) {
      issue_w((rank * KS + s * 128) * N, slot);
      LAS char* lx = (LAS char*)smem + slot * SLOT + lw * 32 * 128;
#pragma unroll
      for (int i = 0; i < 4; ++i)
        __builtin_amdgcn_raw_ptr_buffer_load_lds(rxl, lx + i * 1024, 16, (i & 1) ? vx1 : vx0, s * 128 + i * 8 * KS, 0, 0);
    };
    const bool pfw = PF_ON && lw >= 2;
    const int pf_voff = (4 * idx + (lane >> 4)) * N + grp * 2048 + (lane & 15) * 128;
    auto prefetch = [&](int u) {
      int p, q; piece_pq(u < NREM ? u : NREM - 1, rank, p, q);
      __builtin_amdgcn_raw_ptr_buffer_load_lds(rw, (LAS char*)smem + LDS_SCRAP, 4, pf_voff, (p * KS + q * 128) * N, 0, 0);
    };
    auto prefetch_step = [&](int t) {
      if (t <= PF_T2) prefetch(1 + 2 * t + (lw & 1));
      else if (((t ^ lw) & 1) == 0) prefetch(2 + PF_T2 + t);
    };
    auto wait_slot = [&](int t) {
      if (!pfw) asm volatile("s_waitcnt vmcnt(12)" ::: "memory");
      else if (t >= 2 && (t <= PF_T2 + 1 || (t == PF_T2 + 2 && ((t ^ lw) & 1)))) asm volatile("s_waitcnt vmcnt(14)" ::: "memory");
      else asm volatile("s_waitcnt vmcnt(13)" ::: "memory");
    };
    auto wait_first = [&]() {
      if (PF_ON && lw == 3) asm volatile("s_waitcnt vmcnt(13)" ::: "memory");
      else asm volatile("s_waitcnt vmcnt(12)" ::: "memory");
    };
    issue_local(0, 0); issue_local(1, 1);
    if (PF_ON && lw == 3) prefetch(0);
    if (lw == 0 && lane < 40 && lane != 1) lctl[lane] = 0u;
    wait_first();
    asm volatile("s_waitcnt lgkmcnt(0)" ::: "memory");
    BAR();
    issue_local(2, 2);
    const unsigned call = __builtin_amdgcn_readfirstlane(lctl[1]);
    __amdgpu_buffer_rsrc_t rxs = __builtin_amdgcn_make_buffer_rsrc((void*)(myws + WS_X + (size_t)(call & 1u) * NR * BLK), 0, NR * BLK, 0x00020000);
    __amdgpu_buffer_rsrc_t rfl = __builtin_amdgcn_make_buffer_rsrc((void*)(myws + WS_FLAG + mq * 1024), 0, 1024, 0x00020000);
    unsigned flg_addr, my_entry;
    {
      int p, q; piece_pq(min(lane >> 1, NREM - 1), rank, p, q);
      flg_addr = (unsigned)(size_t)((LAS char*)smem + LDS_FLG) + (p * 4 + q) * 32 + (lane & 1) * 16;
      my_entry = 0x100u | (unsigned)(p * 4 + q);
    }
    auto poll_issue = [&]() {
      __builtin_amdgcn_raw_ptr_buffer_load_lds(rfl, (LAS char*)smem + LDS_FLG, 16, lane * 16, 0, 0, 17);
      __builtin_amdgcn_raw_ptr_buffer_load_lds(rfl, (LAS char*)smem + LDS_SCRAP, 4, 0x7ff00000, 0, 0, 17);
    };
    auto poll_eval = [&]() -> unsigned long long {
      const v4u f0 = *(volatile LAS v4u*)(size_t)flg_addr;
      const unsigned m = min(min(f0.x, f0.y), min(f0.z, f0.w));
      const unsigned long long ok = __ballot(m >= call && lane < 2 * NREM);
      return ok & (ok >> 1) & 0x5555555555555555ull;
    };
    unsigned long long ready = 0, used = 0;
    constexpr unsigned long long ALL = 0x0055555555555555ull;
    int nsched = 4;
    auto schedule = [&](int upto) {
      const unsigned long long avail = ready & ~used;
      const int q0 = (__builtin_ctzll(ALL & ~used) >> 1) / 7;
      const unsigned long long gq = 0x1555ull << (14 * q0);
      const unsigned long long first = ready == ALL ? avail : (avail & gq), rest = avail & ~first;
      const int nfirst = __builtin_popcountll(first);
      const bool in_first = (first >> lane) & 1ull;
      const unsigned long long m = in_first ? first : rest;
      const int pos = nsched + (in_first ? 0 : nfirst) + (int)__builtin_amdgcn_mbcnt_hi((unsigned)(m >> 32), __builtin_amdgcn_mbcnt_lo((unsigned)m, 0u));
      const bool take = ((avail >> lane) & 1ull) && (in_first || pos <= upto);
      if (take) lsch[pos] = my_entry;
      const unsigned long long taken = __ballot(take);
      used |= taken;
      nsched += __builtin_popcountll(taken);
    };
    auto issue_remote = [&](unsigned e, int slot) {
      const int pq = (int)(e & 31u);
      issue_w(pq * 128 * N, slot);
      LAS char* lx = (LAS char*)smem + slot * SLOT + lw * 32 * 128;
      const int sox = pq * (MS * 128);
#pragma unroll
      for (int i = 0; i < 4; ++i)
        __builtin_amdgcn_raw_ptr_buffer_load_lds(rxs, lx + i * 1024, 16, (i & 1) ? vs1 : vs0, sox + i * 1024, 0, 17);
    };
    wait_first();
    BAR();
    int poll_age = 0;
    issue_local(3, 0);
    if (lw == 0 && ready != ALL) { poll_issue(); poll_age = 1; }
    if (pfw) prefetch_step(0);
    int slot = 1;
    int t = 1;
    unsigned tbl = 0;
    for (; t < NSTEP - 3; ++t) {
      if (poll_age == 1 || poll_age == 2) asm volatile("s_waitcnt vmcnt(14)" ::: "memory");
      else wait_slot(t);
      const int s = t + 3;
      if (lw == 0) {
        if (poll_age == 2) {
          unsigned sts; asm volatile("s_getreg_b32 %0, hwreg(HW_REG_IB_STS)" : "=s"(sts));
          if (((sts & 15u) | ((sts >> 18) & 0x30u)) <= 12u) { asm volatile("s_waitcnt vmcnt(12)" ::: "memory"); poll_age = 3; }
        }
        bool news = false;
        if (poll_age == 3) { ready |= poll_eval(); poll_age = 0; news = true; }
        if ((news || nsched <= s + 2) && (ready & ~used)) schedule(s + 2);
      }
      unsigned e = __builtin_amdgcn_readlane(tbl, s);
      BAR();
      if (!e) {
        tbl = lsch[lane & 31];
        e = __builtin_amdgcn_readlane(tbl, s);
        if (!e) {
          if (lw == 0) {
            const unsigned long long t0 = __builtin_amdgcn_s_memrealtime();
            for (unsigned n = 1;; ++n) {
              if (n > 1 || poll_age == 0) poll_issue();
              asm volatile("s_waitcnt vmcnt(0)" ::: "memory");
              ready |= poll_eval();
              if (ready & ~used) break;
              if ((n & 15u) == 0u) {
                const unsigned tv = __builtin_amdgcn_readfirstlane(__hip_atomic_load(tmo, __ATOMIC_RELAXED, __HIP_MEMORY_SCOPE_AGENT));
                if (tv == call || __builtin_amdgcn_s_memrealtime() - t0 > DEADLINE_TICKS) {
                  if (lane == 0) { __hip_atomic_store(tmo, call, __ATOMIC_RELAXED, __HIP_MEMORY_SCOPE_AGENT); lctl[0] = 1u; }
                  ready = ALL;
                  break;
                }
              }
            }
            poll_age = 0;
            schedule(s + 2);
          }
          for (;;) {
            tbl = lsch[lane & 31];
            e = __builtin_amdgcn_readlane(tbl, s);
            if (e) break;
            __builtin_amdgcn_s_sleep(1);
          }
        }
      }
      const unsigned tbl_next = lsch[lane & 31];
      const int have = (lw == 0 && ready != ALL && poll_age == 0) ? nsched - s - 1 + __builtin_popcountll(ready & ~used) : 99;
      if (have < POLL_URGENT) poll_issue();
      issue_remote(e, slot);
      if (pfw) prefetch_step(t);
      if (poll_age) ++poll_age;
      else if (have < POLL_URGENT) poll_age = 3;
      else if (have < POLL_LOW || (have < 99 && (t & POLL_EVERY) == 0)) { poll_issue(); poll_age = 1; }
      tbl = tbl_next;
      slot = (slot == 2) ? 0 : slot + 1;
      if (__builtin_amdgcn_readlane(tbl, NSTEP - 1) != 0u && poll_age == 0) { ++t; break; }
    }
    for (; t < NSTEP - 3; ++t) {
      if (t < PF_T2 + 3) wait_slot(t);
      else if (pfw) asm volatile("s_waitcnt vmcnt(13)" ::: "memory");
      else asm volatile("s_waitcnt vmcnt(12)" ::: "memory");
      const unsigned e = __builtin_amdgcn_readlane(tbl, t + 3);
      BAR();
      issue_remote(e, slot);
      if (pfw) prefetch_step(t);
      slot = (slot == 2) ? 0 : slot + 1;
    }
    if (lw == 0 && lane == 0) {
      __amdgpu_buffer_rsrc_t rt = __builtin_amdgcn_make_buffer_rsrc((void*)tmo, 0, 4, 0x00020000);
      __builtin_amdgcn_raw_ptr_buffer_load_lds(rt, (LAS char*)smem + LDS_CTRL + 8, 4, 0, 0, 0, 16);
    }
    wait_slot(29);
    BAR();
    asm volatile("s_waitcnt vmcnt(0)" ::: "memory");
    BAR();
    BAR();
    return;
  }

  const int dst = (grp + idx) & 7;
  const bool pushes = (dst != rank);
  v4i px0, px1;
  const int poff = (idx * 16 + (lane >> 3)) * KS + wid * 128 + (lane & 7) * 16;
  char* const dws = P.ws[dst];
  const float sx0 = P.sx[0], sw0 = P.sw[0];
  unsigned call;
  {
    const char* src = P.x + (size_t)dst * BLK + poff;
    if (wid == 0) {
      unsigned* cp = (unsigned*)(myws + WS_CTR) + b * 32;
      unsigned tick = 0, one = 1u;
      asm volatile("s_mov_b64 exec, 1\n\tglobal_atomic_add %0, %3, %4, off sc0\n\ts_mov_b64 exec, -1\n\t"
                   "global_load_dwordx4 %1, %5, off\n\tglobal_load_dwordx4 %2, %6, off\n\ts_waitcnt vmcnt(0)"
                   : "+v"(tick), "=&v"(px0), "=&v"(px1) : "v"(cp), "v"(one), "v"(src), "v"(src + 8 * KS) : "memory");
      call = __builtin_amdgcn_readfirstlane(tick) + 1u;
      if (lane == 0) lctl[1] = call;
    } else {
      asm volatile("global_load_dwordx4 %0, %2, off\n\tglobal_load_dwordx4 %1, %3, off\n\ts_waitcnt vmcnt(0)"
                   : "=&v"(px0), "=&v"(px1) : "v"(src), "v"(src + 8 * KS) : "memory");
      call = 0;
    }
  }
  const float scale = __builtin_bit_cast(float, __builtin_amdgcn_readfirstlane(__builtin_bit_cast(int, sx0 * sw0)));
  char* const pdst0 = dws + WS_X + (size_t)rank * BLK + wid * (MS * 128) + (idx * 16 + (lane >> 3)) * 128 + (lane & 7) * 16;
  unsigned* const pflag = (unsigned*)(dws + WS_FLAG) + (((idx >> 3) * 8 + rank) * 4 + wid) * 8 + (idx & 7);
  auto push_stores = [&]() {
    char* const pdst = pdst0 + (size_t)(call & 1u) * NR * BLK;
    asm volatile("global_store_dwordx4 %0, %1, off sc0 sc1\n\ts_nop 1" :: "v"(pdst), "v"(px0) : "memory");
    asm volatile("global_store_dwordx4 %0, %1, off offset:1024 sc0 sc1\n\ts_nop 1" :: "v"(pdst), "v"(px1) : "memory");
  };
  const int t_store = T_STORE(wid);
  const int t_force = t_store + 3;
  int pstate = pushes ? 0 : 2;
  if (pushes && t_store < 0) { push_stores(); pstate = 1; }
  if (pstate == 1) { asm volatile("s_waitcnt vmcnt(0)" ::: "memory"); if (lane == 0) __hip_atomic_store(pflag, call, __ATOMIC_RELAXED, __HIP_MEMORY_SCOPE_SYSTEM); pstate = 2; }
  int t_evt = pstate == 2 ? NSTEP : (pstate == 1 ? 0 : t_store);
  auto push_event = [&](int t, bool force) {
    if (pstate == 0) { push_stores(); pstate = 1; }
    else {
      unsigned sts; asm volatile("s_getreg_b32 %0, hwreg(HW_REG_IB_STS)" : "=s"(sts));
      if (force || (sts & 0x00C0000Fu) == 0u) {
        asm volatile("s_waitcnt vmcnt(0)" ::: "memory");
        if (lane == 0) __hip_atomic_store(pflag, call, __ATOMIC_RELAXED, __HIP_MEMORY_SCOPE_SYSTEM);
        pstate = 2; t_evt = NSTEP;
      }
    }
  };

  const int wm = wid >> 1, wn = wid & 1;
  const int g = lane >> 4, l15 = lane & 15;
  const int fx = (l15 >> 1) & 7;
  const unsigned ax0 = (wm * 64 + l15) * 128 + ((g ^ fx) << 4);
  const unsigned ax1 = (wm * 64 + l15) * 128 + (((4 + g) ^ fx) << 4);
  const int q8 = l15 >> 1;
  const unsigned aw0 = XB + (16 * g + q8) * 256 + (((8 * wn) ^ (q8 | ((g & 1) << 3))) << 4) + 8 * (lane & 1);
  v4f acc[4][8];
#pragma unroll
  for (int i = 0; i < 4; ++i)
#pragma unroll
    for (int j = 0; j < 8; ++j) acc[i][j] = (v4f){};
  v8i af[4], bl[4], bh[4];
  const unsigned sb0 = (unsigned)(size_t)(LAS char*)smem;
#define RDA(i, a0_, a1_) af[i] = cat4(*(LAS v4i*)(size_t)((a0_) + (i) * 2048), *(LAS v4i*)(size_t)((a1_) + (i) * 2048))
#define TR(t, aw_) __builtin_amdgcn_ds_read_tr8_b64_v2i32((LAS v2i*)(size_t)((aw_) + ((t) >> 1) * 16384 + ((t) & 1) * 2048))
#define RDB(dst, j, aw_) dst[(j) & 3] = cat2(TR(0, (aw_) ^ ((j) << 4)), TR(1, (aw_) ^ ((j) << 4)), TR(2, (aw_) ^ ((j) << 4)), TR(3, (aw_) ^ ((j) << 4)))
#define MM(i, j, B) acc[i][j] = __builtin_amdgcn_mfma_scale_f32_16x16x128_f8f6f4(af[i], B[(j) & 3], acc[i][j], 1, 1, 0, 0, 0, 0)
#define MM4(i, j0, B) do { MM(i, j0, B); MM(i, j0 + 1, B); MM(i, j0 + 2, B); MM(i, j0 + 3, B); } while (0)
#define SGB(m, n) __builtin_amdgcn_sched_group_barrier(m, n, 0)
#define P1(n) SGB(0x8,1); SGB(0x100,n);
  WAIT_LGKM0();
  BAR();
  call = __builtin_amdgcn_readfirstlane(lctl[1]);
  {
    const unsigned w0 = aw0 + sb0;
    RDB(bl, 0, w0); RDB(bl, 1, w0); RDB(bl, 2, w0); RDB(bl, 3, w0);
    RDA(0, ax0 + sb0, ax1 + sb0); RDA(1, ax0 + sb0, ax1 + sb0); RDA(2, ax0 + sb0, ax1 + sb0);
  }
  unsigned xc0 = ax0 + sb0, xc1 = ax1 + sb0, wc = aw0 + sb0;
  unsigned sbn = sb0 + SLOT;
  char* const outp = (char*)P.out;
  asm volatile("" :: "s"(outp));
  SCHED();
  WAIT_LGKM0();
#pragma nounroll
  for (int t = 0; t < NSTEP; ++t) {
    SCHED();
    RDA(3, xc0, xc1);
    MM(0, 0, bl);
    P1(2)
    SCHED();
    if (t >= t_evt) push_event(t, t >= t_force);
    SCHED();
    RDB(bh, 4, wc); RDB(bh, 5, wc); RDB(bh, 6, wc); RDB(bh, 7, wc);
    const unsigned xn0 = ax0 + sbn, xn1 = ax1 + sbn, wn = aw0 + sbn;
    MM(0, 1, bl); MM(0, 2, bl); MM(0, 3, bl);
    MM4(1, 0, bl); MM4(2, 0, bl); MM4(3, 0, bl);
    P1(2) P1(2) P1(2) P1(2) P1(1) P1(1) P1(1) P1(1) P1(1) P1(1) P1(1) P1(1) SGB(0x8, 3);
    SCHED();
    WAIT_LGKM0();
    BAR();
    SCHED();
    RDB(bl, 0, wn); RDB(bl, 1, wn);
    MM4(0, 4, bh);
    P1(2) P1(2) P1(2) P1(2)
    SCHED();
    RDA(0, xn0, xn1); RDB(bl, 2, wn);
    MM4(1, 4, bh);
    P1(2) P1(2) P1(1) P1(1)
    SCHED();
    RDA(1, xn0, xn1); RDB(bl, 3, wn);
    MM4(2, 4, bh);
    P1(2) P1(2) P1(1) P1(1)
    SCHED();
    RDA(2, xn0, xn1);
    xc0 = xn0; xc1 = xn1; wc = wn;
    sbn = sbn == sb0 + 2 * SLOT ? sb0 : sbn + SLOT;
    MM4(3, 4, bh);
    P1(1) P1(1) SGB(0x8, 2);
    SCHED();
  }
  const bool bad = (lctl[2] == call) || (lctl[0] != 0u);
  const float s = bad ? __builtin_nanf("") : scale;
  constexpr float LOG2E = 1.44269504088896f, ZMAX = 60.0f * LOG2E;
  const v2f s2 = {s, s}, c2 = {-s * LOG2E, -s * LOG2E};
  __amdgpu_buffer_rsrc_t ro = __builtin_amdgcn_make_buffer_rsrc((void*)outp, 0, MS * N * 4, 0x00020000);
  const int vo = ((m0 + wm * 64 + 8 * (lane >> 5)) * N + n0 + wn * 128 + (lane & 31)) * 4;
  auto silu2 = [&](float a0, float a1) -> v2f {
    const v2f a = {a0, a1};
    const v2f y = a * s2, z = a * c2;
    const v2f d = (v2f){__builtin_amdgcn_exp2f(__builtin_amdgcn_fmed3f(z.x, -ZMAX, ZMAX)), __builtin_amdgcn_exp2f(__builtin_amdgcn_fmed3f(z.y, -ZMAX, ZMAX))} + 1.0f;
    return y * (v2f){__builtin_amdgcn_rcpf(d.x), __builtin_amdgcn_rcpf(d.y)};
  };
#pragma unroll
  for (int i = 0; i < 4; ++i)
#pragma unroll
    for (int rp = 0; rp < 4; rp += 2)
#pragma unroll
      for (int jp = 0; jp < 4; ++jp) {
        const v2f u = silu2(acc[i][2 * jp][rp], acc[i][2 * jp][rp + 1]), v = silu2(acc[i][2 * jp + 1][rp], acc[i][2 * jp + 1][rp + 1]);
        const float u0 = u.x, u1 = u.y, v0 = v.x, v1 = v.y;
        const auto w0 = __builtin_amdgcn_permlane16_swap(__float_as_uint(u0), __float_as_uint(v0), false, false);
        const auto w1 = __builtin_amdgcn_permlane16_swap(__float_as_uint(u1), __float_as_uint(v1), false, false);
        __builtin_amdgcn_raw_buffer_store_b32(w0[0], ro, vo + jp * 128, (16 * i + rp) * N * 4, OUT_AUX);
        __builtin_amdgcn_raw_buffer_store_b32(w0[1], ro, vo + jp * 128, (16 * i + rp + 4) * N * 4, OUT_AUX);
        __builtin_amdgcn_raw_buffer_store_b32(w1[0], ro, vo + jp * 128, (16 * i + rp + 1) * N * 4, OUT_AUX);
        __builtin_amdgcn_raw_buffer_store_b32(w1[1], ro, vo + jp * 128, (16 * i + rp + 5) * N * 4, OUT_AUX);
      }
}

extern "C" void kernel_launch(int rank, int nranks, ncclComm_t comm,
                              hipStream_t stream,
                              void* const* d_in, const int* in_sizes, int n_in,
                              void* d_out, int out_size,
                              const int* in_dtypes, int out_dtype,
                              void* const* d_out_all,
                              void* const* d_ws_all, size_t ws_bytes) {
  (void)comm; (void)in_dtypes; (void)out_dtype; (void)d_out_all;
  if (nranks != NR || rank < 0 || rank >= NR || n_in != 4 || in_sizes[0] != NR * BLK || in_sizes[1] != K * N ||
      out_size != MS * N || ws_bytes < WS_NEED) {
    fprintf(stderr, "kernel_launch: unexpected problem shape\n");
    return;
  }
  static bool attr_set = false;
  if (!attr_set) {
    HIP_CHECK(hipFuncSetAttribute((const void*)a2a_gemm_k, hipFuncAttributeMaxDynamicSharedMemorySize, LDS_BYTES));
    attr_set = true;
  }
  Params P;
  P.x = (const char*)d_in[0]; P.w = (const char*)d_in[1]; P.sx = (const float*)d_in[2]; P.sw = (const float*)d_in[3];
  P.out = (float*)d_out; P.myws = (char*)d_ws_all[rank];
  for (int p = 0; p < NR; ++p) P.ws[p] = (char*)d_ws_all[p];
  P.rank = rank; P.pad = 0;
  a2a_gemm_k<<<256, 512, LDS_BYTES, stream>>>(P);
  HIP_CHECK(hipGetLastError());
}
```
